```python
import numpy as np

kernel_name = "gpu_gemm_rs_m4096_k4096_n16384_e4m3e5m2_dqsilu"


ABSMAX_RTOL = 0.05

COMPUTE_DTYPE = "float32"
EPILOGUE = "dequant_silu"
IN_QUANT = None
OUT_QUANT = None
IN_FP8 = "e4m3e5m2"
IN_DTYPES = {"x": "e4m3", "w_mat": "e5m2",
             "scale_x": "float32", "scale_w": "float32"}
WORLD_SIZE = 8

def _snap_fp8(x, mant_bits, max_abs, min_normal):
    a = np.abs(x.astype(np.float64))
    _m, _e = np.frexp(a)
    q = np.where(a >= min_normal, np.ldexp(1.0, _e - 1 - mant_bits),
                 min_normal * 2.0 ** -mant_bits)
    snapped = np.minimum(np.round(a / q) * q, max_abs)
    return (np.sign(x) * snapped).astype(np.float32)

def setup_inputs():

    x = _snap_fp8(np.random.randn(4096, 4096).astype(np.float32), 3, 448.0, 2.0 ** -6)
    w_mat = _snap_fp8(np.random.randn(4096, 16384).astype(np.float32), 2, 57344.0, 2.0 ** -14)
    scale_x = (0.02 * (1.0 + np.random.rand(1))).astype(np.float32)
    scale_w = (0.02 * (1.0 + np.random.rand(1))).astype(np.float32)
    return {"x": x, "w_mat": w_mat, "scale_x": scale_x, "scale_w": scale_w}

def reference(x, w_mat, scale_x, scale_w):
    acc = (x @ w_mat).astype(np.float32)
    y = (acc * (scale_x * scale_w)).astype(np.float32)
    return (y / (1.0 + np.exp(-np.clip(y, -60.0, 60.0)))).astype(np.float32)
```

```cpp
#include <hip/hip_runtime.h>
#include <rccl/rccl.h>
#include <cstdio>
#include <cstdint>

#define HIP_CHECK(expr) do { hipError_t _e = (expr); if (_e != hipSuccess) { \
    fprintf(stderr, "HIP error %s at %s:%d\n", hipGetErrorString(_e), __FILE__, __LINE__); \
    return; } } while (0)

typedef int      v8i __attribute__((ext_vector_type(8)));
typedef int      v4i __attribute__((ext_vector_type(4)));
typedef unsigned v4u __attribute__((ext_vector_type(4)));
typedef unsigned v2u __attribute__((ext_vector_type(2)));
typedef float    v4f __attribute__((ext_vector_type(4)));

namespace {

constexpr int NR = 8;
constexpr int GN = 16384;
constexpr int KSH = 512;
constexpr int NSH = 2048;
constexpr int NWG = 256;
constexpr int NSEND = 32;
constexpr int NKS = 32;

constexpr int T_WA = 0, T_X0 = 64, T_WB = 120, T_X1 = 184, T_XO = 352, T_OUT = 384, T_END = 896;
constexpr int NCLS = 7;

constexpr unsigned OFF_TMO   = 128;
constexpr unsigned OFF_TICK  = 1024;
constexpr unsigned OFF_RCNT  = 2048;
constexpr unsigned OFF_RLIST = 3072;
constexpr unsigned OFF_IN    = 4096;
constexpr unsigned OFF_DONE  = 4096 + 8192;
constexpr unsigned OFF_CALL  = 32768;
constexpr unsigned OFF_DBG   = 65536;
constexpr unsigned OFF_STAGE = 1u << 20;
constexpr unsigned X_BYTES = 16u << 20, W_BYTES = 8u << 20, STAGE_BYTES = X_BYTES + W_BYTES;
constexpr unsigned OFF_OST = OFF_STAGE + 2 * STAGE_BYTES;
constexpr unsigned OST_BYTES = 32u << 20;
constexpr size_t   WS_NEED = (size_t)OFF_OST + OST_BYTES;
constexpr unsigned OUT_BYTES = 512u * GN * 4u;

constexpr unsigned long long DEADLINE_TICKS = 50000000ull;

constexpr int AUX_SC1 = 16;
constexpr int LDS_STAGE = 48 * 1024;
constexpr int LDS_BYTES = 3 * LDS_STAGE + 64;

struct Params {
  const uint8_t* x;
  const uint8_t* w;
  const float* sx;
  const float* sw;
  float* out[NR];
  uint8_t* ws[NR];
  int rank;
};

template <int N> __device__ __forceinline__ void wait_vm() { asm volatile("s_waitcnt vmcnt(%0)" :: "i"(N) : "memory"); }
__device__ __forceinline__ void raw_barrier() {
  asm volatile("s_waitcnt lgkmcnt(0)" ::: "memory");
  __builtin_amdgcn_s_barrier();
  asm volatile("" ::: "memory");
}
__device__ __forceinline__ unsigned rfl(unsigned v) { return __builtin_amdgcn_readfirstlane(v); }
typedef __attribute__((address_space(1))) uint8_t gu8;
typedef __attribute__((address_space(1))) unsigned gu32;
__device__ __forceinline__ gu8* rfl_ptr(const void* ptr) {
  const unsigned long long a = (unsigned long long)ptr;
  return (gu8*)(((unsigned long long)rfl((unsigned)(a >> 32)) << 32) | rfl((unsigned)a));
}
__device__ __forceinline__ v4i make_rsrc(const gu8* base, unsigned bytes) {
  const unsigned long long a = (unsigned long long)base;
  v4i r;
  r[0] = (int)rfl((unsigned)a);
  r[1] = (int)(rfl((unsigned)(a >> 32)) & 0xffffu);
  r[2] = (int)bytes;
  r[3] = 0x00020000;
  return r;
}

__device__ __forceinline__ bool wait_words(gu32* addr, bool active, unsigned target, gu32* tmo, unsigned call) {
  const unsigned long long t0 = __builtin_amdgcn_s_memrealtime();
  for (;;) {
    unsigned v = target;
    if (active) v = __hip_atomic_load(addr, __ATOMIC_RELAXED, __HIP_MEMORY_SCOPE_SYSTEM);
    if (__all((int)(v - target) >= 0)) return true;
    if (rfl(__hip_atomic_load(tmo, __ATOMIC_RELAXED, __HIP_MEMORY_SCOPE_AGENT)) == call ||
        __builtin_amdgcn_s_memrealtime() - t0 > DEADLINE_TICKS) {
      __hip_atomic_store(tmo, call, __ATOMIC_RELAXED, __HIP_MEMORY_SCOPE_AGENT);
      return false;
    }
    __builtin_amdgcn_s_sleep(4);
  }
}

#define STAMP_AT(i) do {} while (0)

__device__ __forceinline__ void sender(const Params& p, const int b, const int wave, const int lane) {
  const int rank = (int)rfl((unsigned)p.rank);
  const int q = (rank + 1 + wave) & (NR - 1);
  const unsigned nt = wave == 7 ? T_OUT : T_END;
  gu8* const myws = rfl_ptr(p.ws[rank]);
  gu8* const dws = rfl_ptr(p.ws[q]);
  gu32* const tmo = (gu32*)(myws + OFF_TMO);
  const int g = lane >> 4, r = lane & 15;
  const unsigned zero = 0, one = 1;

  unsigned callv = 0;
  {
    gu32* cp = (gu32*)(myws + OFF_CALL + (b * 8 + wave) * 16);
    if (lane == 0) asm volatile("s_nop 4\n\tglobal_atomic_add %0, %1, %2, %3 sc0" : "=v"(callv) : "v"(zero), "v"(one), "s"(cp) : "memory");
  }
  const v4i rs_w = make_rsrc(rfl_ptr(p.w), KSH * GN);
  const v4i rs_x = make_rsrc(rfl_ptr(p.x), 4096 * KSH);
  const v4i rs_o = make_rsrc(myws + OFF_OST + q * (4u << 20), 4u << 20);
  const v4i rd_o = make_rsrc(rfl_ptr(p.out[q]), OUT_BYTES);
  gu32* const tick = (gu32*)(myws + OFF_TICK + wave * 128);
  gu32* const rlist = (gu32*)(myws + OFF_RLIST + q * 128);
  gu32* const inflag = (gu32*)(dws + OFF_IN + rank * 1024);

  const unsigned w_voff = (unsigned)((((lane >> 3) & 3) * 16 + (lane >> 5) * 8) * GN + (lane & 7) * 16 + q * NSH);
  const unsigned w_soff = (unsigned)((((lane >> 3) & 3) * 16 + (lane & 7) * 2 + (lane >> 5)) * 16);
  const unsigned x_voff = (unsigned)(r * KSH + g * 16);
  const unsigned l16 = lane * 16;

  v4u A[8], B[8];
  auto is_w = [](unsigned t) { return t < T_X0 || (t >= T_WB && t < T_X1); };
  auto issue_loads = [&](unsigned t, unsigned tile, bool valid) {
    unsigned so, str, vo;
    v4i rs;
    if (is_w(t)) {
      const unsigned wt = t < T_X0 ? t : t - T_WB + 64;
      so = ((wt >> 2) & 3) * (128u * GN) + ((wt >> 1) & 1) * (64u * GN) + (wt >> 4) * 256 + (wt & 1) * 128;
      str = GN; vo = w_voff; rs = rs_w;
    } else if (t < T_OUT) {
      unsigned row;
      if (t >= T_XO) row = q * 512 + (t - T_XO) * 16;
      else {
        const unsigned xt = t < T_WB ? t - T_X0 : t - T_X1 + 56;
        const unsigned s = xt / 56, u = xt % 56;
        row = ((q + 1 + (u >> 3)) & 7) * 512 + s * 128 + (u & 7) * 16;
      }
      so = row * KSH; str = 64; vo = x_voff; rs = rs_x;
    } else {
      so = tile * (128u << 10) + ((t - T_OUT) & 15) * 8192;
      str = 1024; vo = l16; rs = rs_o;
    }
    if (!valid) { rs[2] = 0; so = 0; }
#pragma unroll
    for (int k = 0; k < 4; ++k) rs[k] = (int)rfl((unsigned)rs[k]);
    so = rfl(so); str = rfl(str);
    asm volatile("s_nop 4" ::: "memory");
#pragma unroll
    for (int e = 0; e < 8; ++e)
      asm volatile("buffer_load_dwordx4 %0, %1, %2, %3 offen sc1" : "=v"(B[e]) : "v"(vo), "s"(rs), "s"(so + e * str) : "memory");
  };
  auto pinB = [&]() {
#pragma unroll
    for (int e = 0; e < 8; ++e) asm volatile("" : "+v"(B[e]));
  };
  auto promote = [&](unsigned t) {
    if (is_w(t)) {
      unsigned R[16][2];
#pragma unroll
      for (int k = 0; k < 8; ++k)
#pragma unroll
        for (int e = 0; e < 2; ++e) {
          const auto sw = __builtin_amdgcn_permlane32_swap(B[k][e], B[k][2 + e], false, false);
          R[k][e] = sw[0]; R[8 + k][e] = sw[1];
        }
#pragma unroll
      for (int u = 0; u < 4; ++u)
#pragma unroll
        for (int e = 0; e < 2; ++e) {
          const unsigned a0 = R[4 * u][e], a1 = R[4 * u + 1][e], a2 = R[4 * u + 2][e], a3 = R[4 * u + 3][e];
          const unsigned t0 = __builtin_amdgcn_perm(a1, a0, 0x05010400u), t1 = __builtin_amdgcn_perm(a1, a0, 0x07030602u);
          const unsigned t2 = __builtin_amdgcn_perm(a3, a2, 0x05010400u), t3 = __builtin_amdgcn_perm(a3, a2, 0x07030602u);
          A[4 * e + 0][u] = __builtin_amdgcn_perm(t2, t0, 0x05040100u);
          A[4 * e + 1][u] = __builtin_amdgcn_perm(t2, t0, 0x07060302u);
          A[4 * e + 2][u] = __builtin_amdgcn_perm(t3, t1, 0x05040100u);
          A[4 * e + 3][u] = __builtin_amdgcn_perm(t3, t1, 0x07060302u);
        }
    } else {
#pragma unroll
      for (int e = 0; e < 8; ++e) A[e] = B[e];
    }
  };

  constexpr unsigned UNKNOWN = 0xffffffffu;
  unsigned t0 = b, t1 = b + NSEND, t2 = b + 2 * NSEND;
  unsigned tile0 = 0, tile1 = 0;
  issue_loads(t0, 0, true);
  wait_vm<0>();
  pinB();
  asm volatile("" : "+v"(callv));
  const unsigned call = rfl(callv) + 1;
  const unsigned tag = call & 0xffffffu;
  const unsigned tbase = (call - 1) * nt - 3 * NSEND;
  const v4i rd_s = make_rsrc(dws + OFF_STAGE + (call & 1) * STAGE_BYTES, STAGE_BYTES);
  promote(t0);
  unsigned nout = 0;
  bool alive = true;
  while (t0 < nt) {
    unsigned dso, st_hi, st_lo;
    v4i rd = rd_s;
    unsigned cls = 0, svo = l16;
    if (is_w(t0)) {
      svo = w_soff;
      const unsigned wt = t0 < T_X0 ? t0 : t0 - T_WB + 64;
      dso = X_BYTES + ((wt >> 4) * NKS + rank * 4 + ((wt >> 2) & 3)) * 32768 + (wt & 1) * 16384 + ((wt >> 1) & 1) * 1024;
      st_hi = 4096; st_lo = 2048;
      cls = t0 < T_X0 ? 0 : 2;
    } else if (t0 < T_OUT) {
      unsigned rg, mb;
      if (t0 >= T_XO) { rg = q * 4 + ((t0 - T_XO) >> 3); mb = (t0 - T_XO) & 7; cls = 6; }
      else {
        const unsigned xt = t0 < T_WB ? t0 - T_X0 : t0 - T_X1 + 56;
        const unsigned s = xt / 56, u = xt % 56;
        rg = ((q + 1 + (u >> 3)) & 7) * 4 + s; mb = u & 7; cls = s == 0 ? 1 : 2 + s;
      }
      dso = (rg * NKS + rank * 4) * 16384 + mb * 2048;
      st_hi = 16384; st_lo = 1024;
    } else {
      rd = rd_o;
      dso = (((tile0 >> 3) * 128 + ((t0 - T_OUT) & 15) * 8) * GN + rank * NSH + (tile0 & 7) * 256) * 4;
      st_hi = 2 * GN * 4; st_lo = GN * 4;
      ++nout;
    }
    dso = rfl(dso); st_hi = rfl(st_hi); st_lo = rfl(st_lo);
#pragma unroll
    for (int k = 0; k < 4; ++k) rd[k] = (int)rfl((unsigned)rd[k]);
    auto st2 = [&](int gi) {
      asm volatile("buffer_store_dwordx4 %0, %1, %2, %3 offen sc0 sc1" :: "v"(A[2 * gi]), "v"(svo), "s"(rd), "s"(dso + gi * st_hi) : "memory");
      asm volatile("buffer_store_dwordx4 %0, %1, %2, %3 offen sc0 sc1\n\ts_nop 1" :: "v"(A[2 * gi + 1]), "v"(svo), "s"(rd), "s"(dso + gi * st_hi + st_lo) : "memory");
    };
    asm volatile("s_nop 4" ::: "memory");
    st2(0);
    unsigned rv = 0, tk = 0;
    const bool l1 = t1 < nt;
    const bool early = l1 && tile1 != UNKNOWN;
    issue_loads(t1, tile1, early);
    {
      const unsigned slot = t2 >= T_OUT && t2 < T_END ? (t2 - T_OUT) >> 4 : 0;
      asm volatile("global_load_dword %0, %1, %2 sc1" : "+v"(rv) : "v"(slot * 4), "s"(rlist) : "memory");
      if (lane == 0) asm volatile("global_atomic_add %0, %1, %2, %3 sc0" : "+v"(tk) : "v"(zero), "v"(one), "s"(tick) : "memory");
    }
    wait_vm<10>();
    st2(1);
    wait_vm<0>();
    st2(2);
    wait_vm<0>();
    st2(3);
    wait_vm<0>();
    pinB();
    asm volatile("" : "+v"(rv), "+v"(tk));
    if (t0 < T_OUT && lane == 0)
      asm volatile("global_atomic_add %0, %1, %2 sc1" :: "v"(cls * 128), "v"(one), "s"(inflag) : "memory");
    if (l1 && !early) {
      const unsigned slot = (t1 - T_OUT) >> 4;
      const unsigned long long c0 = __builtin_amdgcn_s_memrealtime();
      for (;;) {
        const unsigned e = rfl(__hip_atomic_load(rlist + slot, __ATOMIC_RELAXED, __HIP_MEMORY_SCOPE_AGENT));
        if ((e >> 8) == tag) { tile1 = e & 31; break; }
        if (rfl(__hip_atomic_load(tmo, __ATOMIC_RELAXED, __HIP_MEMORY_SCOPE_AGENT)) == call ||
            __builtin_amdgcn_s_memrealtime() - c0 > DEADLINE_TICKS) {
          __hip_atomic_store(tmo, call, __ATOMIC_RELAXED, __HIP_MEMORY_SCOPE_AGENT);
          alive = false; tile1 = 0;
          break;
        }
        __builtin_amdgcn_s_sleep(2);
      }
      issue_loads(t1, tile1, true);
      wait_vm<0>();
      pinB();
    }
    promote(t1);
    unsigned tile2 = 0;
    if (t2 >= T_OUT && t2 < nt) {
      const unsigned e = rfl(rv);
      tile2 = (e >> 8) == tag ? (e & 31) : UNKNOWN;
    }
    t0 = t1; tile0 = tile1;
    t1 = t2; tile1 = tile2;
    t2 = rfl(tk) - tbase;
  }
  if (wave != 7) {
    if (alive && lane == 0)
      __hip_atomic_fetch_add((gu32*)(dws + OFF_DONE + rank * 128), nout, __ATOMIC_RELAXED, __HIP_MEMORY_SCOPE_SYSTEM);
  } else if (b == 0) {
    const bool fin = wait_words((gu32*)(myws + OFF_DONE + (lane & 7) * 128), lane < 8 && lane != rank,
                                call * (unsigned)(T_END - T_OUT), tmo, call);
    if (!fin && lane == 0) p.out[rank][rank * NSH] = __builtin_nanf("");
  }
}

__global__ void __launch_bounds__(512, 2) fused_kernel(Params p) {
  __shared__ __attribute__((aligned(16))) uint8_t smem[LDS_BYTES];
  const int tid = threadIdx.x;
  const int lane = tid & 63;
  const int wave = __builtin_amdgcn_readfirstlane(tid >> 6);
  const int b = blockIdx.x;
  STAMP_AT(0);
  if (b < NSEND) {
    sender(p, b, wave, lane);
    STAMP_AT(7);
    return;
  }
  const int rank = p.rank;
  uint8_t* const myws = p.ws[rank];
  gu32* const tmo = (gu32*)(myws + OFF_TMO);
  const int g = lane >> 4, r = lane & 15;
  volatile __attribute__((address_space(3))) unsigned* const bc =
      (volatile __attribute__((address_space(3))) unsigned*)(smem + 3 * LDS_STAGE);

  if (tid == 0) {
    const unsigned t = __hip_atomic_fetch_add((unsigned*)(myws + OFF_CALL + b * 128), 1u, __ATOMIC_RELAXED, __HIP_MEMORY_SCOPE_AGENT);
    bc[0] = t + 1;
  }
  __syncthreads();
  const unsigned call = rfl(bc[0]);
  const int nb = b & 7, u = (b - NSEND) >> 3;
  const int wm = wave >> 2, wn = wave & 3;
  const float scale = p.sx[0] * p.sw[0];

  for (int pass = 0; pass < 2; ++pass) {
    if (pass == 1 && u >= 4) break;
    const int stage = pass == 0 ? u / 7 : u;
    const int po = pass == 0 ? (rank + 1 + u % 7) & (NR - 1) : rank;
    if (wave == 0) {
      const int src = lane & 7;
      const int xcls = pass == 1 ? 6 : (stage == 0 ? 1 : 2 + stage);
      const int cls = (lane & 8) ? xcls : (nb < 4 ? 0 : 2);
      const unsigned cnt = (lane & 8) ? (pass == 1 ? 32u : 56u) : 64u;
      gu32* f = (gu32*)(myws + OFF_IN + src * 1024 + cls * 128);
      const bool okw = wait_words(f, lane < 16, call * cnt, tmo, call);
      __builtin_amdgcn_fence(__ATOMIC_ACQUIRE, "");
      asm volatile("s_waitcnt vmcnt(0)" ::: "memory");
      if (lane == 0) bc[1] = okw ? 1u : 0u;
    }
    __syncthreads();
    const bool ok = rfl(bc[1]) != 0;
    STAMP_AT(1 + pass * 3);

    v4f acc[4][4];
#pragma unroll
    for (int i = 0; i < 4; ++i)
#pragma unroll
      for (int j = 0; j < 4; ++j) acc[i][j] = (v4f){0.f, 0.f, 0.f, 0.f};

    if (ok) {
      const uint8_t* stg = myws + OFF_STAGE + (size_t)(call & 1) * STAGE_BYTES;
      const uint8_t* xa = stg + (size_t)((po * 4 + stage) * NKS) * 16384 + wave * 1024 + lane * 16;
      const uint8_t* wb = stg + X_BYTES + (size_t)(nb * NKS) * 32768 + wave * 1024 + lane * 16;
      auto issue = [&](int ks, int sb) {
        uint8_t* l = smem + sb * LDS_STAGE + wave * 1024;
        const uint8_t* a = xa + (size_t)ks * 16384;
        const uint8_t* w = wb + (size_t)ks * 32768;
#pragma unroll
        for (int i = 0; i < 2; ++i)
          __builtin_amdgcn_global_load_lds((const __attribute__((address_space(1))) void*)(a + i * 8192),
                                           (__attribute__((address_space(3))) void*)(l + i * 8192), 16, 0, 0);
#pragma unroll
        for (int i = 0; i < 4; ++i)
          __builtin_amdgcn_global_load_lds((const __attribute__((address_space(1))) void*)(w + i * 8192),
                                           (__attribute__((address_space(3))) void*)(l + 16384 + i * 8192), 16, 0, 0);
      };
      auto compute = [&](int sb) {
        const uint8_t* la = smem + sb * LDS_STAGE + (wm * 4) * 2048 + lane * 16;
        const uint8_t* lb = smem + sb * LDS_STAGE + 16384 + (wn * 4) * 2048 + lane * 16;
        v8i af[4], bf[4];
#pragma unroll
        for (int i = 0; i < 4; ++i) {
          const v4i lo = *(const v4i*)(la + i * 2048), hi = *(const v4i*)(la + i * 2048 + 1024);
          af[i] = __builtin_shufflevector(lo, hi, 0, 1, 2, 3, 4, 5, 6, 7);
        }
#pragma unroll
        for (int j = 0; j < 4; ++j) {
          const v4i lo = *(const v4i*)(lb + j * 2048), hi = *(const v4i*)(lb + j * 2048 + 1024);
          bf[j] = __builtin_shufflevector(lo, hi, 0, 1, 2, 3, 4, 5, 6, 7);
        }
#pragma unroll
        for (int i = 0; i < 4; ++i)
#pragma unroll
          for (int j = 0; j < 4; ++j)
            acc[i][j] = __builtin_amdgcn_mfma_scale_f32_16x16x128_f8f6f4(af[i], bf[j], acc[i][j],   0,   1,
                                                                          0, 0x7f7f7f7f, 0, 0x7f7f7f7f);
      };
      issue(0, 0);
      issue(1, 1);
      int sb = 0;
      for (int ks = 0; ks < NKS - 2; ++ks) {
        wait_vm<6>();
        raw_barrier();
        issue(ks + 2, sb == 0 ? 2 : sb - 1);
        compute(sb);
        sb = sb == 2 ? 0 : sb + 1;
      }
      wait_vm<6>();
      raw_barrier();
      compute(sb);
      sb = sb == 2 ? 0 : sb + 1;
      wait_vm<0>();
      raw_barrier();
      compute(sb);
    }
    STAMP_AT(2 + pass * 3);

    raw_barrier();
    {
      uint8_t* lo = smem + (wm * 64 + g * 4) * 1024 + (wn >> 1) * 512 + r * 32 + (wn & 1) * 16;
      const float qnan = __builtin_nanf("");
#pragma unroll
      for (int i = 0; i < 4; ++i)
#pragma unroll
        for (int reg = 0; reg < 4; ++reg) {
          v4f o;
#pragma unroll
          for (int j = 0; j < 4; ++j) {
            const float y = acc[i][j][reg] * scale;
            const float yc = __builtin_amdgcn_fmed3f(y, -60.f, 60.f);
            const float e = __builtin_amdgcn_exp2f(yc * -1.4426950408889634f);
            o[j] = ok ? y * __builtin_amdgcn_rcpf(1.f + e) : qnan;
          }
          *(v4f*)(lo + (i * 16 + reg) * 1024) = o;
        }
    }
    raw_barrier();
    {
      const int tile = stage * 8 + nb;
      const __amdgpu_buffer_rsrc_t ro = pass == 0
          ? __builtin_amdgcn_make_buffer_rsrc(myws + OFF_OST, 0, OST_BYTES, 0x00020000)
          : __builtin_amdgcn_make_buffer_rsrc(p.out[rank], 0, OUT_BYTES, 0x00020000);
      const unsigned obase = pass == 0 ? (unsigned)((po * 32 + tile) * (128 << 10) + wave * 16384 + lane * 16)
                                       : (unsigned)(((stage * 128 + wave * 16) * GN + rank * NSH + nb * 256) * 4 + lane * 16);
      const unsigned ostr = pass == 0 ? 1024u : (unsigned)(GN * 4);
      const uint8_t* lrow = smem + (wave * 16) * 1024 + lane * 16;
#pragma unroll
      for (int t = 0; t < 16; ++t) {
        const v4u v = *(const v4u*)(lrow + t * 1024);
        __builtin_amdgcn_raw_buffer_store_b128(v, ro, obase + (unsigned)t * ostr, 0, AUX_SC1);
      }
      asm volatile("s_waitcnt vmcnt(0)" ::: "memory");
      __syncthreads();
      if (tid == 0 && pass == 0) {
        const unsigned slot = __hip_atomic_fetch_add((unsigned*)(myws + OFF_RCNT + po * 128), 1u, __ATOMIC_RELAXED,
                                                     __HIP_MEMORY_SCOPE_AGENT) & 31u;
        __hip_atomic_store((unsigned*)(myws + OFF_RLIST + po * 128) + slot, (call << 8) | (unsigned)tile, __ATOMIC_RELAXED,
                           __HIP_MEMORY_SCOPE_AGENT);
      }
    }
    STAMP_AT(3 + pass * 3);
  }
}

}

extern "C" void kernel_launch(int rank, int nranks, ncclComm_t comm,
                              hipStream_t stream,
                              void* const* d_in, const int* in_sizes, int n_in,
                              void* d_out, int out_size,
                              const int* in_dtypes, int out_dtype,
                              void* const* d_out_all,
                              void* const* d_ws_all, size_t ws_bytes) {
  (void)comm; (void)in_dtypes; (void)out_dtype; (void)d_out;
  if (nranks != NR || n_in != 4 || ws_bytes < WS_NEED || out_size != 512 * GN ||
      in_sizes[0] != 4096 * KSH || in_sizes[1] != KSH * GN || (unsigned)rank >= (unsigned)NR) {
    fprintf(stderr, "kernel_launch: unexpected problem configuration\n");
    return;
  }
  Params p;
  p.x = (const uint8_t*)d_in[0];
  p.w = (const uint8_t*)d_in[1];
  p.sx = (const float*)d_in[2];
  p.sw = (const float*)d_in[3];
  for (int i = 0; i < NR; ++i) {
    p.out[i] = (float*)d_out_all[i];
    p.ws[i] = (uint8_t*)d_ws_all[i];
  }
  p.rank = rank;
  hipLaunchKernelGGL(fused_kernel, dim3(NWG), dim3(512), 0, stream, p);
  HIP_CHECK(hipGetLastError());
}
```

```cpp
#include <hip/hip_runtime.h>
#include <rccl/rccl.h>
#include <cstdio>
#include <cstdint>

#define HIP_CHECK(expr) do { hipError_t _e = (expr); if (_e != hipSuccess) { \
    fprintf(stderr, "HIP error %s at %s:%d\n", hipGetErrorString(_e), __FILE__, __LINE__); \
    return; } } while (0)

typedef int      v8i __attribute__((ext_vector_type(8)));
typedef int      v4i __attribute__((ext_vector_type(4)));
typedef unsigned v4u __attribute__((ext_vector_type(4)));
typedef float    v4f __attribute__((ext_vector_type(4)));

namespace {

constexpr int NR = 8;
constexpr int GN = 16384;
constexpr int KSH = 512;
constexpr int NSH = 2048;
constexpr int NWG = 256;
#define NSEND_ 64
constexpr int NSEND = NSEND_;
constexpr int NG = (NWG - NSEND) / 8;
#define NWL 28
constexpr int NKS = 32;

constexpr unsigned T_W = 224, T_X1 = 736, T_XO = 1408, T_OUT = 1536, T_END = 3584;

constexpr unsigned OFF_TMO   = 128;
constexpr unsigned OFF_TICK  = 1024;
constexpr unsigned OFF_RCNT  = 2048;
constexpr unsigned OFF_IN    = 4096;
constexpr unsigned OFF_DONE  = 8192;
constexpr unsigned OFF_SNAP  = 12288;
constexpr unsigned OFF_CALL  = 32768;
constexpr unsigned OFF_MASK  = 65536;
constexpr unsigned OFF_DBG   = 131072;
constexpr unsigned OFF_STAGE = 1u << 20;
constexpr unsigned SRC_BYTES = T_OUT * 2048, STAGE_BYTES = NR * SRC_BYTES, W_BYTES = 8u << 20;
constexpr unsigned OFF_OST = OFF_STAGE + 2 * STAGE_BYTES;
constexpr unsigned OST_BYTES = 32u << 20;
constexpr unsigned OFF_WT = OFF_OST + OST_BYTES;
constexpr unsigned WS_NEED = OFF_WT + W_BYTES;
constexpr unsigned OUT_BYTES = 512u * GN * 4u;

constexpr unsigned long long DEADLINE_TICKS = 50000000ull;

#define WAITN 5
#define ACQ 2
#define ACQ_INSN asm volatile("" ::: "memory")
#define DMA_AUX 17
constexpr int AUX_SC1 = 16;
constexpr int LDS_STAGE = 48 * 1024;
constexpr int LDS_BYTES = 3 * LDS_STAGE + 256 + 8 * 256;

struct Params {
  const uint8_t* x;
  const uint8_t* w;
  const float* sx;
  const float* sw;
  float* out[NR];
  uint8_t* ws[NR];
  int rank;
};

typedef __attribute__((address_space(1))) uint8_t gu8;
typedef __attribute__((address_space(1))) unsigned gu32;

template <int N> __device__ __forceinline__ void wait_vm() { asm volatile("s_waitcnt vmcnt(%0)" :: "i"(N) : "memory"); }
__device__ __forceinline__ void raw_barrier() {
  asm volatile("s_waitcnt lgkmcnt(0)" ::: "memory");
  __builtin_amdgcn_s_barrier();
  asm volatile("" ::: "memory");
}
__device__ __forceinline__ unsigned rfl(unsigned v) { return __builtin_amdgcn_readfirstlane(v); }
__device__ __forceinline__ gu8* rfl_ptr(const void* ptr) {
  const unsigned long long a = (unsigned long long)ptr;
  return (gu8*)(((unsigned long long)rfl((unsigned)(a >> 32)) << 32) | rfl((unsigned)a));
}
__device__ __forceinline__ v4i make_rsrc(unsigned long long a, unsigned bytes) {
  v4i r;
  r[0] = (int)rfl((unsigned)a);
  r[1] = (int)(rfl((unsigned)(a >> 32)) & 0xffffu);
  r[2] = (int)rfl(bytes);
  r[3] = 0x00020000;
  return r;
}
__device__ __forceinline__ bool timed_out(gu32* tmo, unsigned call, unsigned long long t0, unsigned& spins) {
  if (((++spins & 63u) == 0 && rfl(__hip_atomic_load(tmo, __ATOMIC_RELAXED, __HIP_MEMORY_SCOPE_AGENT)) == call) ||
      __builtin_amdgcn_s_memrealtime() - t0 > DEADLINE_TICKS) {
    __hip_atomic_store(tmo, call, __ATOMIC_RELAXED, __HIP_MEMORY_SCOPE_AGENT);
    return true;
  }
  return false;
}

#define STAMP_AT(i) do {} while (0)

__device__ __forceinline__ void monitor(const Params& p, gu8* myws, int rank, unsigned call, int lane) {
  gu32* const tmo = (gu32*)(myws + OFF_TMO);
  const unsigned long long t0 = __builtin_amdgcn_s_memrealtime();
  unsigned spins = 0;
  bool dead = false;
  {
    const int grp = lane >> 3;
    const bool act = grp < 6;
    const gu8* f = myws + OFF_IN + (lane & 7) * 128 + (act ? grp : 0) * 16;
    const unsigned tg = call * (grp < 2 ? 64u : 56u);
    const unsigned tgt[4] = {grp == 5 ? call * 128u : tg, grp == 5 ? call * 224u : tg, grp == 5 ? 0u : tg, grp == 5 ? 0u : tg};
    unsigned pub = 0;
    for (;;) {
      v4u v;
      asm volatile("global_load_dwordx4 %0, %1, off sc0 sc1\n\ts_waitcnt vmcnt(0)" : "=v"(v) : "v"(f) : "memory");
      unsigned mask = 0;
#pragma unroll
      for (int k = 0; k < 4; ++k) {
        const unsigned long long miss = __ballot(act && (int)(v[k] - tgt[k]) < 0);
#pragma unroll
        for (int gq = 0; gq < 6; ++gq)
          if (((miss >> (8 * gq)) & 0xff) == 0) mask |= 1u << (4 * gq + k);
      }
      if (mask != pub) {
        pub = mask;
        const unsigned word = (call << 24) | mask;
#pragma unroll
        for (int i = 0; i < 4; ++i)
          __hip_atomic_store((gu32*)(myws + OFF_MASK + (i * 64 + lane) * 128), word, __ATOMIC_RELAXED, __HIP_MEMORY_SCOPE_AGENT);
      }
      if (mask == 0xffffffu) break;
      if (timed_out(tmo, call, t0, spins)) { dead = true; break; }
    }
  }
  gu32* f = (gu32*)(myws + OFF_DONE + (lane & 7) * 128);
  const bool act = lane < 8 && lane != rank;
  const unsigned target = call * (T_END - T_OUT);
  while (!dead) {
    unsigned v = target;
    if (act) v = __hip_atomic_load(f, __ATOMIC_RELAXED, __HIP_MEMORY_SCOPE_SYSTEM);
    if (__all((int)(v - target) >= 0)) break;
    if (timed_out(tmo, call, t0, spins)) dead = true;
  }
  if (dead && lane == 0) p.out[rank][rank * NSH] = __builtin_nanf("");
}

constexpr unsigned CLS_OUT = (OFF_DONE - OFF_IN) / 4;
struct Piece {
  unsigned kind;
  unsigned so, dso;
  unsigned cls;
};
__device__ __forceinline__ unsigned xpos(unsigned q, unsigned rg) {
  const unsigned o = rg >> 2, st = rg & 3;
  return o == q ? T_XO * 2048 + st * 16384 : ((st * 28 + ((o - q - 1) & 7)) * 8 + (st ? T_X1 - 224 : 0)) * 2048;
}
__device__ __forceinline__ unsigned xstride(unsigned q, unsigned rg) { return (rg >> 2) == q ? 4u * 16384 : 7u * 16384; }
__device__ __forceinline__ unsigned wpos(unsigned nb) { return (T_W + (nb >> 2) * 256 + (nb & 3) * 16) * 2048; }

__device__ __forceinline__ Piece decode(unsigned t, unsigned tile, unsigned rank, unsigned q, unsigned xs) {
  Piece k;
  k.dso = rank * SRC_BYTES + t * 2048;
  if (t >= T_OUT) {
    const unsigned sub = (t - T_OUT) & 63;
    k.kind = 2; k.cls = CLS_OUT;
    k.so = OFF_OST + (q * 32 + tile) * (128u << 10) + sub * 2048;
    k.dso = (((tile >> 3) * 128 + sub * 2) * GN + rank * NSH + (tile & 7) * 256) * 4;
  } else if (t >= T_W && t < T_X1) {
    k.kind = 1; k.cls = (t - T_W) >> 6;
    k.so = OFF_WT + q * (1u << 20) + (t - T_W) * 2048;
  } else {
    unsigned rg, ksl, mb;
    if (t >= T_XO) { const unsigned y = t - T_XO; ksl = y >> 5; rg = q * 4 + ((y >> 3) & 3); mb = y & 7; k.cls = 20; }
    else {
      const unsigned y = t < T_W ? t : t - T_X1 + 224;
      const unsigned sk = y / 56, z = y % 56;
      ksl = sk & 3; rg = ((q + 1 + (z >> 3)) & 7) * 4 + (sk >> 2); mb = z & 7;
      k.cls = sk < 4 ? 21 : 4 + sk;
    }
    k.kind = xs ? 1 : 0;
    k.so = xs ? xs + xpos(rank, rg) + ksl * xstride(rank, rg) + mb * 2048 : (rg * 128 + mb * 16) * KSH + ksl * 128;
  }
  return k;
}

__device__ __forceinline__ void sender(const Params& p, const int b, const int wave, const int lane) {
  const int rank = (int)rfl((unsigned)p.rank);
  const int gw = wave * NSEND + b;
  gu8* const myws = rfl_ptr(p.ws[rank]);
  const unsigned one = 1;
  if (gw >= 7 * NWL && gw != NSEND * 8 - 1) return;

  if (gw == NSEND * 8 - 1) {
    unsigned c = 0;
    if (lane == 0) c = __hip_atomic_fetch_add((gu32*)(myws + OFF_CALL + gw * 16), 1u, __ATOMIC_RELAXED, __HIP_MEMORY_SCOPE_AGENT);
    monitor(p, myws, rank, rfl(c) + 1, lane);
    return;
  }
  gu32* const tmo = (gu32*)(myws + OFF_TMO);
  const int link = gw % 7, idx = gw / 7;
  const int q = (rank + 1 + link) & (NR - 1);
  const unsigned long long a_my = (unsigned long long)myws, a_x = (unsigned long long)rfl_ptr(p.x);
  const unsigned long long a_d = (unsigned long long)rfl_ptr(p.ws[q]), a_o = (unsigned long long)rfl_ptr(p.out[q]);
  const unsigned x_voff = (unsigned)((lane & 15) * KSH + (lane >> 4) * 16), l16 = lane * 16;
  const unsigned l0off = lane == 0 ? 0u : 0xfffffff0u;
  const unsigned snap_off = OFF_SNAP + q * 256, tick_off = OFF_TICK + link * 128, flag_off = OFF_IN + rank * 128;
  const unsigned l4 = lane * 4;

  v4u D[3][2];
  unsigned SN[3] = {0, 0, 0}, TK[3] = {0, 0, 0};
  auto loads = [&](const Piece k, v4u (&d)[2]) {
    const v4i rs = make_rsrc(k.kind == 0 ? a_x : a_my, k.kind == 3 ? 0u : (k.kind == 0 ? 4096u * KSH : WS_NEED));
    const unsigned vo = k.kind == 0 ? x_voff : l16;
    const unsigned so = rfl(k.kind == 3 ? 0u : k.so), so2 = rfl(so + (k.kind == 0 ? 64u : 1024u));
    asm volatile("s_nop 4\n\tbuffer_load_dwordx4 %0, %1, %2, %3 offen sc1" : "=v"(d[0]) : "v"(vo), "s"(rs), "s"(so) : "memory");
    asm volatile("buffer_load_dwordx4 %0, %1, %2, %3 offen sc1" : "=v"(d[1]) : "v"(vo), "s"(rs), "s"(so2) : "memory");
  };
  Piece k0 = decode(idx, 0, rank, q, 0), k1 = decode(idx + NWL, 0, rank, q, 0);
  const v4i rs_my = make_rsrc(a_my, 1u << 20);
  unsigned callv = one;
  asm volatile("s_nop 4\n\tbuffer_atomic_add %0, %1, %2, %3 offen sc0" : "+v"(callv) : "v"(l0off), "s"(rs_my), "s"(OFF_CALL + gw * 16) : "memory");
  loads(k0, D[0]);
  loads(k1, D[1]);
  wait_vm<0>();
  asm volatile("" : "+v"(callv), "+v"(D[0][0]), "+v"(D[0][1]), "+v"(D[1][0]), "+v"(D[1][1]));
  const unsigned call = rfl(callv) + 1;
  const unsigned tag = call & 0xffffffu;
  const unsigned tbase = (call - 1) * (T_END - 2 * NWL) - 4 * NWL;
  const unsigned long long a_s = a_d + OFF_STAGE + (call & 1) * STAGE_BYTES;
  const unsigned xself = OFF_STAGE + (call & 1) * STAGE_BYTES + rank * SRC_BYTES;
  unsigned f0 = idx + 2 * NWL, f1 = idx + 3 * NWL, fcnt = 2;
  unsigned req1 = 0, req2 = 0;
  unsigned snap = 0;
  unsigned pcls = 0xff;
  unsigned acls = 0, acnt = 0;
  unsigned spins = 63;
  unsigned long long tw = 0;
  auto iter = [&](auto ii) {
    constexpr int c = decltype(ii)::value, n1 = (c + 1) % 3, n2 = (c + 2) % 3;
    {
      const v4i rd = make_rsrc(k0.kind == 2 ? a_o : a_s, k0.kind == 3 ? 0u : (k0.kind == 2 ? OUT_BYTES : STAGE_BYTES));
      const unsigned d0 = rfl(k0.kind == 3 ? 0u : k0.dso), d1 = rfl(d0 + (k0.kind == 2 ? GN * 4u : 1024u));
      asm volatile("s_nop 4\n\tbuffer_store_dwordx4 %0, %1, %2, %3 offen sc0 sc1" :: "v"(D[c][0]), "v"(l16), "s"(rd), "s"(d0) : "memory");
      asm volatile("buffer_store_dwordx4 %0, %1, %2, %3 offen sc0 sc1\n\ts_nop 1" :: "v"(D[c][1]), "v"(l16), "s"(rd), "s"(d1) : "memory");
    }
    snap = SN[n1];
    if (req2) { const unsigned t = rfl(TK[n1]) - tbase; if (fcnt == 0) f0 = t; else f1 = t; ++fcnt; }
    {
      const bool flush = acnt != 0 && (WAITN == 5 ? k0.cls : pcls) != acls;
      const v4i rf = make_rsrc(a_d, flush ? 1u << 20 : 0u);
      const unsigned fo = rfl(flag_off + acls * 4);
      asm volatile("s_nop 4\n\tbuffer_atomic_add %0, %1, %2, %3 offen sc1" :: "v"(acnt), "v"(l0off), "s"(rf), "s"(fo) : "memory");
      acnt = flush ? 0u : acnt;
    }
    Piece k2;
    k2.kind = 3; k2.cls = 0xff; k2.so = 0; k2.dso = 0;
    bool take = false;
    if (f0 < T_END) {
      unsigned tile = 0;
      take = true;
      if (f0 >= T_OUT || (f0 >= T_W && f0 < T_X1)) {
        const unsigned e = __builtin_amdgcn_readlane(snap, f0 >= T_OUT ? (f0 - T_OUT) >> 6 : 32 + ((f0 - T_W) >> 5));
        take = (e >> 8) == tag;
        tile = e & 31;
      }
      if (take) {
        const unsigned xs = __builtin_amdgcn_readlane(snap, 48) == call * 32 ? xself : 0u;
        k2 = decode(f0, tile, rank, q, xs);
        f0 = f1; --fcnt; tw = 0;
      }
      else {
        if (tw == 0) tw = __builtin_amdgcn_s_memrealtime();
        else if (k0.kind == 3 && k1.kind == 3 && timed_out(tmo, call, tw, spins)) f0 = T_END;
      }
    }
    loads(k2, D[n2]);
    asm volatile("buffer_load_dword %0, %1, %2, %3 offen sc1" : "=v"(SN[c]) : "v"(l4), "s"(rs_my), "s"(snap_off) : "memory");
    {
      const v4i rt = make_rsrc(a_my, take ? 1u << 20 : 0u);
      TK[c] = one;
      asm volatile("s_nop 4\n\tbuffer_atomic_add %0, %1, %2, %3 offen sc0" : "+v"(TK[c]) : "v"(l0off), "s"(rt), "s"(tick_off) : "memory");
    }
    wait_vm<WAITN>();
    asm volatile("" : "+v"(D[n1][0]), "+v"(D[n1][1]), "+v"(SN[n2]), "+v"(TK[n2]));
    {
      const unsigned dc = WAITN == 5 ? k0.cls : pcls;
      acnt += dc != 0xff ? 1u : 0u;
      acls = dc != 0xff ? dc : acls;
      pcls = WAITN == 5 ? 0xffu : k0.cls;
    }
    req2 = req1; req1 = take;
    k0 = k1; k1 = k2;
    return f0 >= T_END && fcnt != 0 && k0.kind == 3 && k1.kind == 3 && acnt == 0 && pcls == 0xff;
  };
  for (;;) {
    if (iter(std::integral_constant<int, 0>{})) break;
    if (iter(std::integral_constant<int, 1>{})) break;
    if (iter(std::integral_constant<int, 2>{})) break;
  }
  wait_vm<0>();
}

__global__ void __launch_bounds__(512, 2) fused_kernel(Params p) {
  __shared__ __attribute__((aligned(16))) uint8_t smem[LDS_BYTES];
  const int tid = threadIdx.x;
  const int lane = tid & 63;
  const int wave = __builtin_amdgcn_readfirstlane(tid >> 6);
  const int b = blockIdx.x;
  STAMP_AT(0);
  if (b < NSEND) {
    sender(p, b, wave, lane);
    return;
  }
  const int rank = (int)rfl((unsigned)p.rank);
  gu8* const myws = rfl_ptr(p.ws[rank]);
  gu32* const tmo = (gu32*)(myws + OFF_TMO);
  const int g = lane >> 4, r = lane & 15;
  volatile __attribute__((address_space(3))) unsigned* const bc =
      (volatile __attribute__((address_space(3))) unsigned*)(smem + 3 * LDS_STAGE);

  if (tid == 0) {
    const unsigned t = __hip_atomic_fetch_add((gu32*)(myws + OFF_CALL + b * 128), 1u, __ATOMIC_RELAXED, __HIP_MEMORY_SCOPE_AGENT);
    bc[0] = t + 1;
  }
  unsigned call;
  if (b >= 128) {
    const int q = (rank + 1 + (b & 7)) & 7, ch = (b - 128) >> 3;
    const int ksl = (ch >> 1) & 3, nbq = (ch >> 3) * 4 + (ch & 1) * 2 + (wave >> 2), wq = wave & 3;
    const int hi = lane >> 5, kb = (lane >> 2) & 7, cg = lane & 3;
    const uint8_t* src = p.w + (size_t)(ksl * 128 + kb * 16 + hi * 8) * GN + q * NSH + nbq * 256 + wq * 64 + cg * 16;
    v4u B[8], A[8];
#pragma unroll
    for (int k = 0; k < 8; ++k) B[k] = *(const v4u*)(src + (size_t)k * GN);
    unsigned R[16][2];
#pragma unroll
    for (int k = 0; k < 8; ++k)
#pragma unroll
      for (int e = 0; e < 2; ++e) {
        const auto sw = __builtin_amdgcn_permlane32_swap(B[k][e], B[k][2 + e], false, false);
        R[k][e] = sw[0]; R[8 + k][e] = sw[1];
      }
#pragma unroll
    for (int u = 0; u < 4; ++u)
#pragma unroll
      for (int e = 0; e < 2; ++e) {
        const unsigned a0 = R[4 * u][e], a1 = R[4 * u + 1][e], a2 = R[4 * u + 2][e], a3 = R[4 * u + 3][e];
        const unsigned t0 = __builtin_amdgcn_perm(a1, a0, 0x05010400u), t1 = __builtin_amdgcn_perm(a1, a0, 0x07030602u);
        const unsigned t2 = __builtin_amdgcn_perm(a3, a2, 0x05010400u), t3 = __builtin_amdgcn_perm(a3, a2, 0x07030602u);
        A[0 + e][u] = __builtin_amdgcn_perm(t2, t0, 0x05040100u);
        A[2 + e][u] = __builtin_amdgcn_perm(t2, t0, 0x07060302u);
        A[4 + e][u] = __builtin_amdgcn_perm(t3, t1, 0x05040100u);
        A[6 + e][u] = __builtin_amdgcn_perm(t3, t1, 0x07060302u);
      }
    __syncthreads();
    call = rfl(bc[0]);
    const unsigned s0 = 16 * ((2 * cg + hi) + 8 * (kb & 1) + 16 * (kb >> 1));
    const unsigned dbase = (q == rank ? OFF_STAGE + (call & 1) * STAGE_BYTES + rank * SRC_BYTES + T_W * 2048
                                      : OFF_WT + q * (1u << 20)) + ch * 65536 + wave * 8192;
    const __amdgpu_buffer_rsrc_t rw = __builtin_amdgcn_make_buffer_rsrc((void*)myws, 0, WS_NEED, 0x00020000);
#pragma unroll
    for (int e = 0; e < 8; ++e)
      __builtin_amdgcn_raw_buffer_store_b128(A[e], rw, dbase + e * 1024 + ((e & 1) ? (s0 ^ 128u) : s0), 0, AUX_SC1);
    asm volatile("s_waitcnt vmcnt(0)" ::: "memory");
    __syncthreads();
    if (tid == 0) {
      if (q == rank)
        __hip_atomic_fetch_add((gu32*)(myws + OFF_IN + rank * 128) + (ch >> 1), 32u, __ATOMIC_RELAXED, __HIP_MEMORY_SCOPE_SYSTEM);
      else
        __hip_atomic_store((gu32*)(myws + OFF_SNAP + q * 256) + 32 + ch, call << 8, __ATOMIC_RELAXED, __HIP_MEMORY_SCOPE_AGENT);
    }
  } else if (b >= 96) {
    const int rg = b - 96, ksl = wave >> 1, hf = wave & 1;
    const uint8_t* src = p.x + (size_t)(rg * 128 + hf * 64 + r) * KSH + ksl * 128 + g * 16;
    v4u B[8];
#pragma unroll
    for (int e = 0; e < 8; ++e) B[e] = *(const v4u*)(src + (e >> 1) * 16 * KSH + (e & 1) * 64);
    __syncthreads();
    call = rfl(bc[0]);
    const unsigned dbase = OFF_STAGE + (call & 1) * STAGE_BYTES + rank * SRC_BYTES + xpos(rank, rg) + ksl * xstride(rank, rg)
                           + hf * 8192 + lane * 16;
    const __amdgpu_buffer_rsrc_t rw = __builtin_amdgcn_make_buffer_rsrc((void*)myws, 0, WS_NEED, 0x00020000);
#pragma unroll
    for (int e = 0; e < 8; ++e) __builtin_amdgcn_raw_buffer_store_b128(B[e], rw, dbase + e * 1024, 0, AUX_SC1);
    asm volatile("s_waitcnt vmcnt(0)" ::: "memory");
    __syncthreads();
    if (tid < 4) {
      const int st = rg & 3;
      const int cls = (rg >> 2) == rank ? 20 : (st == 0 ? 21 : 4 + 4 * st + tid);
      __hip_atomic_fetch_add((gu32*)(myws + OFF_IN + rank * 128) + cls, 8u, __ATOMIC_RELAXED, __HIP_MEMORY_SCOPE_SYSTEM);
    } else if (tid >= 64 && tid < 72) {
      __hip_atomic_fetch_add((gu32*)(myws + OFF_SNAP + (tid - 64) * 256) + 48, 1u, __ATOMIC_RELAXED, __HIP_MEMORY_SCOPE_AGENT);
    }
  } else {
    __syncthreads();
    call = rfl(bc[0]);
  }
  STAMP_AT(7);
  const int nb = b & 7, u = (b - NSEND) >> 3;
  const int wm = wave >> 2, wn = wave & 3;
  const float scale = p.sx[0] * p.sw[0];

#pragma unroll 1
  for (int pass = 0; pass < 2; ++pass) {
    const int item = u + pass * NG;
    if (item >= 32) break;
    const bool own = item >= 28;
    const int stage = own ? item - 28 : item / 7;
    const int po = own ? rank : (rank + 1 + item % 7) & (NR - 1);

    const unsigned wsh = nb < 4 ? 0 : 4;
    const unsigned xbit = own ? 20 : 21, xsh = 4 + 4 * stage;
    const bool xone = own || stage == 0;
    gu32* const fptr = (gu32*)(myws + OFF_MASK + b * 128);
    auto segs_ready = [&](unsigned v) {
      v = rfl(v);
      const unsigned xm = xone ? (((v >> xbit) & 1u) ? 15u : 0u) : (v >> xsh);
      const unsigned m = (v >> 24) == (call & 0xffu) ? ((v >> wsh) & xm & 15u) : 0u;
      return (unsigned)__builtin_ctz(~m);
    };
    auto wait_seg = [&](unsigned seg) {
      const unsigned long long t0 = __builtin_amdgcn_s_memrealtime();
      unsigned n, spins = 63;
      for (;;) {
        n = segs_ready(__hip_atomic_load(fptr, __ATOMIC_RELAXED, __HIP_MEMORY_SCOPE_AGENT));
        if (n > seg) break;
        if (timed_out(tmo, call, t0, spins)) { n = 0x104; break; }
        __builtin_amdgcn_s_sleep(4);
      }
      ACQ_INSN;
      asm volatile("s_waitcnt vmcnt(0)" ::: "memory");
      return n;
    };
    if (wave == 0) {
      const unsigned n = wait_seg(0);
      if (lane == 0) { bc[1] = n; bc[2] = 0; }
    }
    __syncthreads();
    unsigned avail = rfl(bc[1]);
    STAMP_AT(1 + pass * 3);

    v4f acc[4][4];
#pragma unroll
    for (int i = 0; i < 4; ++i)
#pragma unroll
      for (int j = 0; j < 4; ++j) acc[i][j] = (v4f){0.f, 0.f, 0.f, 0.f};

    {
      const gu8* stg = myws + OFF_STAGE + (size_t)(call & 1) * STAGE_BYTES;
      const gu8* xa = stg + xpos(rank, po * 4 + stage) + wave * 1024 + lane * 16;
      const gu8* wb = stg + wpos(nb) + wave * 1024 + lane * 16;
      const unsigned xstr = xstride(rank, po * 4 + stage);
      auto issue = [&](int ks, int sb) {
        uint8_t* l = smem + sb * LDS_STAGE + wave * 1024;
        const gu8* a = xa + (unsigned)(ks & 7) * SRC_BYTES + (unsigned)(ks >> 3) * xstr;
        const gu8* w = wb + (unsigned)(ks & 7) * SRC_BYTES + (unsigned)(ks >> 3) * 131072u;
#pragma unroll
        for (int i = 0; i < 2; ++i)
          __builtin_amdgcn_global_load_lds((const __attribute__((address_space(1))) void*)(a + i * 8192),
                                           (__attribute__((address_space(3))) void*)(l + i * 8192), 16, 0, DMA_AUX);
#pragma unroll
        for (int i = 0; i < 4; ++i)
          __builtin_amdgcn_global_load_lds((const __attribute__((address_space(1))) void*)(w + i * 8192),
                                           (__attribute__((address_space(3))) void*)(l + 16384 + i * 8192), 16, 0, DMA_AUX);
      };
      const int boff = (r & 1) * 1024 + 16 * ((r >> 1) + 8 * ((g & 1) ^ (r & 1)) + 16 * (g >> 1));
      v8i af[4], bf[4];
      auto readf = [&](int sb) {
        const uint8_t* la = smem + sb * LDS_STAGE + (wm * 4) * 2048 + lane * 16;
        const uint8_t* lb = smem + sb * LDS_STAGE + 16384 + (wn * 4) * 2048 + boff;
#pragma unroll
        for (int i = 0; i < 4; ++i) {
          const v4i lo = *(const v4i*)(la + i * 2048), hi = *(const v4i*)(la + i * 2048 + 1024);
          af[i] = __builtin_shufflevector(lo, hi, 0, 1, 2, 3, 4, 5, 6, 7);
        }
#pragma unroll
        for (int j = 0; j < 4; ++j) {
          const v4i lo = *(const v4i*)(lb + j * 2048), hi = *(const v4i*)(lb + j * 2048 + 512);
          bf[j] = __builtin_shufflevector(lo, hi, 0, 1, 2, 3, 4, 5, 6, 7);
        }
      };
      issue(0, 0);
      issue(1, 1);
      wait_vm<6>();
      raw_barrier();
      if (wm == 1) __builtin_amdgcn_s_barrier();
      int sb = 0;
      uint8_t* const mbox = smem + 3 * LDS_STAGE + 256 + wave * 256;
      for (int ks = 0; ks < NKS; ++ks) {
        readf(sb);
        if ((avail & 7) < 4) {
          const int ph = ks & 7;
          if (ph == 4) {
            __builtin_amdgcn_global_load_lds((const __attribute__((address_space(1))) void*)fptr,
                                             (__attribute__((address_space(3))) void*)mbox, 4, 0, AUX_SC1);
          } else if (ph == 5) {
            avail = segs_ready(*(volatile __attribute__((address_space(3))) unsigned*)mbox);
            ACQ_INSN;
          } else if (ph == 6) {
            const unsigned sg = (ks + 2) >> 3;
            if (avail <= sg) avail = wait_seg(sg);
          }
        }
        if (ks + 2 < NKS) { issue(ks + 2, sb == 0 ? 2 : sb - 1); wait_vm<6>(); } else wait_vm<0>();
        raw_barrier();
        __builtin_amdgcn_s_setprio(1);
#pragma unroll
        for (int i = 0; i < 4; ++i)
#pragma unroll
          for (int j = 0; j < 4; ++j)
            acc[i][j] = __builtin_amdgcn_mfma_scale_f32_16x16x128_f8f6f4(af[i], bf[j], acc[i][j],   0,   1,
                                                                          0, 0x7f7f7f7f, 0, 0x7f7f7f7f);
        __builtin_amdgcn_s_setprio(0);
        __builtin_amdgcn_s_barrier();
        asm volatile("" ::: "memory");
        sb = sb == 2 ? 0 : sb + 1;
      }
      if (avail >= 0x100 && lane == 0) bc[2] = 1;
      if (wm == 0) __builtin_amdgcn_s_barrier();
      raw_barrier();
    }
    const bool ok = rfl(bc[2]) == 0;
    STAMP_AT(2 + pass * 3);

    {
      const int tile = stage * 8 + nb;
      const __amdgpu_buffer_rsrc_t ro = !own
          ? __builtin_amdgcn_make_buffer_rsrc((void*)(myws + OFF_OST), 0, OST_BYTES, 0x00020000)
          : __builtin_amdgcn_make_buffer_rsrc(p.out[rank], 0, OUT_BYTES, 0x00020000);
      const unsigned ostr = !own ? 1024u : (unsigned)(GN * 4);
      const unsigned obase = (!own ? (unsigned)((po * 32 + tile) * (128 << 10))
                                        : (unsigned)((stage * 128 * GN + rank * NSH + nb * 256) * 4))
                             + (unsigned)(wm * 64 + g * 4) * ostr + wn * 256 + r * 16;
      const float sc = ok ? scale : __builtin_nanf("");
#pragma unroll
      for (int i = 0; i < 4; ++i)
#pragma unroll
        for (int reg = 0; reg < 4; ++reg) {
          v4f o;
#pragma unroll
          for (int j = 0; j < 4; ++j) {
            const float y = acc[i][j][reg] * sc;
            const float yc = __builtin_amdgcn_fmed3f(y, -60.f, 60.f);
            const float e = __builtin_amdgcn_exp2f(yc * -1.4426950408889634f);
            o[j] = y * __builtin_amdgcn_rcpf(1.f + e);
          }
          __builtin_amdgcn_raw_buffer_store_b128(__builtin_bit_cast(v4u, o), ro, obase + (unsigned)(i * 16 + reg) * ostr, 0, AUX_SC1);
        }
      asm volatile("s_waitcnt vmcnt(0)" ::: "memory");
      __syncthreads();
      if (tid == 0 && !own) {
        const unsigned slot = __hip_atomic_fetch_add((gu32*)(myws + OFF_RCNT + po * 128), 1u, __ATOMIC_RELAXED,
                                                     __HIP_MEMORY_SCOPE_AGENT) & 31u;
        __hip_atomic_store((gu32*)(myws + OFF_SNAP + po * 256) + slot, (call << 8) | (unsigned)tile, __ATOMIC_RELAXED,
                           __HIP_MEMORY_SCOPE_AGENT);
      }
    }
    STAMP_AT(3 + pass * 3);
  }
}

}

extern "C" void kernel_launch(int rank, int nranks, ncclComm_t comm,
                              hipStream_t stream,
                              void* const* d_in, const int* in_sizes, int n_in,
                              void* d_out, int out_size,
                              const int* in_dtypes, int out_dtype,
                              void* const* d_out_all,
                              void* const* d_ws_all, size_t ws_bytes) {
  (void)comm; (void)in_dtypes; (void)out_dtype; (void)d_out;
  if (nranks != NR || n_in != 4 || ws_bytes < (size_t)WS_NEED || out_size != 512 * GN ||
      in_sizes[0] != 4096 * KSH || in_sizes[1] != KSH * GN || (unsigned)rank >= (unsigned)NR) {
    fprintf(stderr, "kernel_launch: unexpected problem configuration\n");
    return;
  }
  Params p;
  p.x = (const uint8_t*)d_in[0];
  p.w = (const uint8_t*)d_in[1];
  p.sx = (const float*)d_in[2];
  p.sw = (const float*)d_in[3];
  for (int i = 0; i < NR; ++i) {
    p.out[i] = (float*)d_out_all[i];
    p.ws[i] = (uint8_t*)d_ws_all[i];
  }
  p.rank = rank;
  hipLaunchKernelGGL(fused_kernel, dim3(NWG), dim3(512), 0, stream, p);
  HIP_CHECK(hipGetLastError());
}
```

```cpp
#include <hip/hip_runtime.h>
#include <rccl/rccl.h>
#include <cstdio>
#include <cstdint>

#define HIP_CHECK(expr) do { hipError_t _e = (expr); if (_e != hipSuccess) { \
    fprintf(stderr, "HIP error %s at %s:%d\n", hipGetErrorString(_e), __FILE__, __LINE__); \
    return; } } while (0)

typedef int      v8i __attribute__((ext_vector_type(8)));
typedef int      v4i __attribute__((ext_vector_type(4)));
typedef unsigned v4u __attribute__((ext_vector_type(4)));
typedef float    v4f __attribute__((ext_vector_type(4)));

namespace {

constexpr int NR = 8;
constexpr int GN = 16384;
constexpr int KSH = 512;
constexpr int NSH = 2048;
constexpr int NWG = 256;
constexpr int NSEND = 64;
constexpr int NG = (NWG - NSEND) / 8;
#define NWL 26
#define NPL 28
#define GATE_LEAD 48
#define GSLEEP 4
#define PH 3
#define XB0 6

constexpr unsigned T_W = XB0 * 48, T_X1 = T_W + 512, T_XO = 512 + 16 * 48, T_OUT = T_XO + 128;

constexpr unsigned SX_W = 16 * 7 * 16384, SX_O = SX_W + (1u << 20), SRC_BYTES = SX_O + 16 * 16384;

constexpr unsigned OFF_TMO   = 128;
constexpr unsigned OFF_TICK  = 1024;
constexpr unsigned OFF_RCNT  = 3072;
constexpr unsigned OFF_IN    = 4096;
constexpr unsigned OFF_SNAP  = 12288;
constexpr unsigned OFF_CALL  = 32768;
constexpr unsigned OFF_MASK  = 65536;
constexpr unsigned OFF_DBG   = 131072;
constexpr unsigned OFF_STAGE = 1u << 20;
constexpr unsigned STAGE_BYTES = NR * SRC_BYTES;
constexpr unsigned OFF_OST = OFF_STAGE + 2 * STAGE_BYTES;
constexpr unsigned OST_BYTES = 32u << 20;
constexpr unsigned OFF_WT = OFF_OST + 2 * OST_BYTES;
constexpr unsigned WS_NEED = OFF_WT + (8u << 20);
constexpr unsigned OUT_BYTES = 512u * GN * 4u;

constexpr unsigned long long DEADLINE_TICKS = 50000000ull;

constexpr int AUX_SC1 = 16;
constexpr int AUX_SYS = 17;
constexpr int LDS_STAGE = 48 * 1024;
constexpr int LDS_BYTES = 3 * LDS_STAGE + 256 + 8 * 256;

struct Params {
  const uint8_t* x;
  const uint8_t* w;
  const float* sx;
  const float* sw;
  float* out;
  uint8_t* ws[NR];
  int rank;
};

typedef __attribute__((address_space(1))) uint8_t gu8;
typedef __attribute__((address_space(1))) unsigned gu32;

template <int N> __device__ __forceinline__ void wait_vm() { asm volatile("s_waitcnt vmcnt(%0)" :: "i"(N) : "memory"); }
__device__ __forceinline__ void raw_barrier() {
  asm volatile("s_waitcnt lgkmcnt(0)" ::: "memory");
  __builtin_amdgcn_s_barrier();
  asm volatile("" ::: "memory");
}
__device__ __forceinline__ unsigned rfl(unsigned v) { return __builtin_amdgcn_readfirstlane(v); }
__device__ __forceinline__ gu8* rfl_ptr(const void* ptr) {
  const unsigned long long a = (unsigned long long)ptr;
  return (gu8*)(((unsigned long long)rfl((unsigned)(a >> 32)) << 32) | rfl((unsigned)a));
}
__device__ __forceinline__ v4i make_rsrc(unsigned long long a, unsigned bytes) {
  v4i r;
  r[0] = (int)rfl((unsigned)a);
  r[1] = (int)(rfl((unsigned)(a >> 32)) & 0xffffu);
  r[2] = (int)rfl(bytes);
  r[3] = 0x00020000;
  return r;
}
__device__ __forceinline__ bool timed_out(gu32* tmo, unsigned call, unsigned long long t0, unsigned& spins) {
  if (((++spins & 63u) == 0 && rfl(__hip_atomic_load(tmo, __ATOMIC_RELAXED, __HIP_MEMORY_SCOPE_AGENT)) == call) ||
      __builtin_amdgcn_s_memrealtime() - t0 > DEADLINE_TICKS) {
    __hip_atomic_store(tmo, call, __ATOMIC_RELAXED, __HIP_MEMORY_SCOPE_AGENT);
    return true;
  }
  return false;
}
__device__ __forceinline__ float silu(float y) {
  const float yc = __builtin_amdgcn_fmed3f(y, -60.f, 60.f);
  const float e = __builtin_amdgcn_exp2f(yc * -1.4426950408889634f);
  return y * __builtin_amdgcn_rcpf(1.f + e);
}

#define STAMP_AT(i) do {} while (0)

__device__ __forceinline__ unsigned xpos(unsigned q, unsigned rg) {
  const unsigned o = rg >> 2, st = rg & 3;
  return o == q ? SX_O + st * 16384 : (st * 28 + ((o - q - 1) & 7)) * 16384;
}
__device__ __forceinline__ unsigned xstride(unsigned q, unsigned rg) { return (rg >> 2) == q ? 4u * 16384 : 7u * 16384; }
__device__ __forceinline__ unsigned wpos(unsigned nb) { return (nb >> 2) * (512u << 10) + (nb & 3) * 32768; }

__device__ __forceinline__ void monitor(gu8* myws, int rank, unsigned call, int lane) {
  gu32* const tmo = (gu32*)(myws + OFF_TMO);
  const unsigned long long t0 = __builtin_amdgcn_s_memrealtime();
  unsigned spins = 0;
  const int grp = lane >> 3;
  const bool act = grp < 6;
  const gu8* f = myws + OFF_IN + (lane & 7) * 128 + (act ? grp : 0) * 16;
  const unsigned xb = (lane & 7) == rank ? 56u : 48u;
  const unsigned tg = call * (grp < 2 ? 64u : xb);
  const unsigned tgt[4] = {grp == 5 ? call * 128u : tg, grp == 5 ? call * 4u * xb : tg, grp == 5 ? 0u : tg, grp == 5 ? 0u : tg};
  unsigned pub = 0;
  for (;;) {
    v4u v;
    asm volatile("global_load_dwordx4 %0, %1, off sc0 sc1\n\ts_waitcnt vmcnt(0)" : "=v"(v) : "v"(f) : "memory");
    unsigned mask = 0;
#pragma unroll
    for (int k = 0; k < 4; ++k) {
      const unsigned long long miss = __ballot(act && (int)(v[k] - tgt[k]) < 0);
#pragma unroll
      for (int gq = 0; gq < 6; ++gq)
        if (((miss >> (8 * gq)) & 0xff) == 0) mask |= 1u << (4 * gq + k);
    }
    if (mask != pub) {
      pub = mask;
      const unsigned word = (call << 24) | mask;
#pragma unroll
      for (int i = 1; i < 4; ++i)
        __hip_atomic_store((gu32*)(myws + OFF_MASK + (i * 64 + lane) * 128), word, __ATOMIC_RELAXED, __HIP_MEMORY_SCOPE_AGENT);
    }
    if (mask == 0xffffffu) break;
    if (timed_out(tmo, call, t0, spins)) break;
  }
}

struct Piece {
  unsigned kind;
  unsigned so, dso;
  unsigned cls;
};
__device__ __forceinline__ Piece decode(unsigned t, unsigned rank, unsigned q, unsigned xs) {
  Piece k;
  if (t >= T_W && t < T_X1) {
    k.kind = 1; k.cls = (t - T_W) >> 6;
    k.so = OFF_WT + q * (1u << 20) + (t - T_W) * 2048;
    k.dso = SX_W + (t - T_W) * 2048;
  } else {
    unsigned rg, ksl, mb;
    if (t >= T_XO) {
      const unsigned y = t - T_XO;
      ksl = y >> 5; rg = q * 4 + ((y >> 3) & 3); mb = y & 7; k.cls = 20;
      k.dso = SX_O + y * 2048;
    } else {
      const unsigned y = t < T_W ? t : t - 512;
      const unsigned sk = y / 48, z = y % 48;
      const unsigned j = z >> 3, rel = j + (j >= ((rank - q - 1) & 7) ? 1u : 0u);
      ksl = sk & 3; rg = ((q + 1 + rel) & 7) * 4 + (sk >> 2); mb = z & 7;
      k.cls = sk < 4 ? 21 : 4 + sk;
      k.dso = (sk * 7 + rel) * 16384 + mb * 2048;
    }
    k.kind = xs ? 1 : 0;
    k.so = xs ? xs + xpos(rank, rg) + ksl * xstride(rank, rg) + mb * 2048 : (rg * 128 + mb * 16) * KSH + ksl * 128;
  }
  return k;
}

__device__ __forceinline__ void pusher(const Params& p, gu8* myws, const int rank, const int gw, const int lane) {
  const unsigned one = 1;
  gu32* const tmo = (gu32*)(myws + OFF_TMO);
  const int link = gw % 7, idx = gw / 7;
  const int q = (rank + 1 + link) & (NR - 1);
  const unsigned long long a_my = (unsigned long long)myws, a_x = (unsigned long long)rfl_ptr(p.x);
  const unsigned long long a_d = (unsigned long long)rfl_ptr(p.ws[q]);
  const unsigned x_voff = (unsigned)((lane & 15) * KSH + (lane >> 4) * 16), l16 = lane * 16;
  const unsigned l0off = lane == 0 ? 0u : 0xfffffff0u;
  const unsigned snap_off = OFF_SNAP + q * 256 + 128, tick_off = OFF_TICK + link * 128, flag_off = OFF_IN + rank * 128;
  const unsigned l4 = (lane & 31) * 4;

  v4u D[3][2];
  unsigned SN[3] = {0, 0, 0}, TK[3] = {0, 0, 0};
  auto loads = [&](const Piece k, v4u (&d)[2]) {
    const v4i rs = make_rsrc(k.kind == 0 ? a_x : a_my, k.kind == 3 ? 0u : (k.kind == 0 ? 4096u * KSH : WS_NEED));
    const unsigned vo = k.kind == 0 ? x_voff : l16;
    const unsigned so = rfl(k.kind == 3 ? 0u : k.so), so2 = rfl(so + (k.kind == 0 ? 64u : 1024u));
    asm volatile("s_nop 4\n\tbuffer_load_dwordx4 %0, %1, %2, %3 offen sc1" : "=v"(d[0]) : "v"(vo), "s"(rs), "s"(so) : "memory");
    asm volatile("buffer_load_dwordx4 %0, %1, %2, %3 offen sc1" : "=v"(d[1]) : "v"(vo), "s"(rs), "s"(so2) : "memory");
  };
  Piece k0 = decode(idx, rank, q, 0), k1 = decode(idx + NWL, rank, q, 0);
  const v4i rs_my = make_rsrc(a_my, 1u << 20);
  unsigned callv = one;
  asm volatile("s_nop 4\n\tbuffer_atomic_add %0, %1, %2, %3 offen sc0" : "+v"(callv) : "v"(l0off), "s"(rs_my), "s"(OFF_CALL + gw * 16) : "memory");
  loads(k0, D[0]);
  loads(k1, D[1]);
  wait_vm<0>();
  asm volatile("" : "+v"(callv), "+v"(D[0][0]), "+v"(D[0][1]), "+v"(D[1][0]), "+v"(D[1][1]));
  const unsigned call = rfl(callv) + 1;
  const unsigned tag = call & 0xffffffu;
  const unsigned tbase = (call - 1) * (T_OUT - 2 * NWL - PH * NPL) - 4 * NWL - PH * NPL;
  const unsigned long long a_s = a_d + OFF_STAGE + (call & 1) * STAGE_BYTES + rank * SRC_BYTES;
  const unsigned xself = OFF_STAGE + (call & 1) * STAGE_BYTES + rank * SRC_BYTES;
  unsigned f0 = idx + 2 * NWL, f1 = idx + 3 * NWL, fcnt = 2;
  unsigned req1 = 0, req2 = 0;
  unsigned acls = 0, acnt = 0;
  unsigned spins = 63;
  unsigned long long tw = 0;
  auto iter = [&](auto ii) {
    constexpr int c = decltype(ii)::value, n1 = (c + 1) % 3, n2 = (c + 2) % 3;
    {
      const v4i rd = make_rsrc(a_s, k0.kind == 3 ? 0u : SRC_BYTES);
      const unsigned d0 = rfl(k0.dso);
      asm volatile("s_nop 4\n\tbuffer_store_dwordx4 %0, %1, %2, %3 offen sc0 sc1" :: "v"(D[c][0]), "v"(l16), "s"(rd), "s"(d0) : "memory");
      asm volatile("buffer_store_dwordx4 %0, %1, %2, %3 offen offset:1024 sc0 sc1\n\ts_nop 1" :: "v"(D[c][1]), "v"(l16), "s"(rd), "s"(d0) : "memory");
      if (d0 == SX_O + (128 - GATE_LEAD) * 2048) {
        const v4i rg = make_rsrc(a_d, 1u << 20);
        asm volatile("s_nop 4\n\tbuffer_store_dword %0, %1, %2, %3 offen sc0 sc1" :: "v"(call), "v"(l0off), "s"(rg), "s"(OFF_SNAP + rank * 256 + 50 * 4) : "memory");
      }
    }
    const unsigned snap = SN[n1];
    if (req2) { const unsigned t = rfl(TK[n1]) - tbase; f0 = fcnt == 0 ? t : f0; f1 = fcnt == 0 ? f1 : t; ++fcnt; }
    {
      const bool flush = acnt != 0 && k0.cls != acls;
      const v4i rf = make_rsrc(a_d, flush ? 1u << 20 : 0u);
      const unsigned fo = rfl(flag_off + acls * 4);
      asm volatile("s_nop 4\n\tbuffer_atomic_add %0, %1, %2, %3 offen sc1" :: "v"(acnt), "v"(l0off), "s"(rf), "s"(fo) : "memory");
      acnt = flush ? 0u : acnt;
    }
    Piece k2;
    k2.kind = 3; k2.cls = 0xff; k2.so = 0; k2.dso = 0;
    bool take = false;
    if (f0 < T_OUT) {
      take = true;
      if (f0 >= T_W && f0 < T_X1) take = (__builtin_amdgcn_readlane(snap, (f0 - T_W) >> 5) >> 8) == tag;
      if (take) {
        const unsigned xs = __builtin_amdgcn_readlane(snap, 16) == call * 32 ? xself : 0u;
        k2 = decode(f0, rank, q, xs);
        f0 = f1; --fcnt; tw = 0;
      } else {
        if (tw == 0) tw = __builtin_amdgcn_s_memrealtime();
        else if (k0.kind == 3 && k1.kind == 3 && timed_out(tmo, call, tw, spins)) f0 = T_OUT;
      }
    }
    loads(k2, D[n2]);
    asm volatile("buffer_load_dword %0, %1, %2, %3 offen sc1" : "=v"(SN[c]) : "v"(l4), "s"(rs_my), "s"(snap_off) : "memory");
    {
      const v4i rt = make_rsrc(a_my, take ? 1u << 20 : 0u);
      TK[c] = one;
      asm volatile("s_nop 4\n\tbuffer_atomic_add %0, %1, %2, %3 offen sc0" : "+v"(TK[c]) : "v"(l0off), "s"(rt), "s"(tick_off) : "memory");
    }
    wait_vm<5>();
    asm volatile("" : "+v"(D[n1][0]), "+v"(D[n1][1]), "+v"(SN[n2]), "+v"(TK[n2]));
    acnt += k0.kind < 3 ? 1u : 0u;
    acls = k0.kind < 3 ? k0.cls : acls;
    req2 = req1; req1 = take;
    k0 = k1; k1 = k2;
    return f0 >= T_OUT && fcnt != 0 && k0.kind == 3 && k1.kind == 3 && acnt == 0;
  };
  for (;;) {
    if (iter(std::integral_constant<int, 0>{})) break;
    if (iter(std::integral_constant<int, 1>{})) break;
    if (iter(std::integral_constant<int, 2>{})) break;
  }
  wait_vm<0>();
}

__device__ __forceinline__ void puller(const Params& p, gu8* myws, const int rank, const int gw, const int lane) {
  constexpr unsigned NPC = 2048, NONE = 0xffffffffu;
  const unsigned one = 1;
  gu32* const tmo = (gu32*)(myws + OFF_TMO);
  const int pw = gw - 7 * NWL, link = pw % 7, idx = pw / 7;
  const int q = (rank + 1 + link) & (NR - 1);
  const unsigned long long a_my = (unsigned long long)myws, a_o = (unsigned long long)rfl_ptr(p.out);
  const unsigned long long a_d = (unsigned long long)rfl_ptr(p.ws[q]);
  const unsigned l16 = lane * 16, l4 = lane * 4;
  const unsigned l0off = lane == 0 ? 0u : 0xfffffff0u;
  const unsigned snap_off = OFF_SNAP + q * 256, tick_off = OFF_TICK + 1024 + link * 128;
  const float scale = p.sx[0] * p.sw[0];
  const v4i rs_my = make_rsrc(a_my, 1u << 20);
  unsigned callv = one;
  asm volatile("s_nop 4\n\tbuffer_atomic_add %0, %1, %2, %3 offen sc0" : "+v"(callv) : "v"(l0off), "s"(rs_my), "s"(OFF_CALL + gw * 16) : "memory");
  unsigned call;
  {
    const v4i rs_x = make_rsrc((unsigned long long)rfl_ptr(p.x), 4096u * KSH);
    const unsigned x_voff = (unsigned)((lane & 15) * KSH + (lane >> 4) * 16);
    v4u H[PH][2];
    unsigned hd[PH];
#pragma unroll
    for (int k = 0; k < PH; ++k) {
      const Piece pc = decode(4 * NWL + idx + k * NPL, rank, q, 0);
      const unsigned o = rfl(pc.so);
      hd[k] = rfl(pc.dso);
      asm volatile("s_nop 4\n\tbuffer_load_dwordx4 %0, %1, %2, %3 offen" : "=v"(H[k][0]) : "v"(x_voff), "s"(rs_x), "s"(o) : "memory");
      asm volatile("buffer_load_dwordx4 %0, %1, %2, %3 offen offset:64" : "=v"(H[k][1]) : "v"(x_voff), "s"(rs_x), "s"(o) : "memory");
    }
    wait_vm<0>();
    asm volatile("" : "+v"(callv));
#pragma unroll
    for (int k = 0; k < PH; ++k) asm volatile("" : "+v"(H[k][0]), "+v"(H[k][1]));
    call = rfl(callv) + 1;
    const v4i rd = make_rsrc(a_d + OFF_STAGE + (call & 1) * STAGE_BYTES + rank * SRC_BYTES, SRC_BYTES);
#pragma unroll
    for (int k = 0; k < PH; ++k) {
      asm volatile("s_nop 4\n\tbuffer_store_dwordx4 %0, %1, %2, %3 offen sc0 sc1" :: "v"(H[k][0]), "v"(l16), "s"(rd), "s"(hd[k]) : "memory");
      asm volatile("buffer_store_dwordx4 %0, %1, %2, %3 offen offset:1024 sc0 sc1\n\ts_nop 1" :: "v"(H[k][1]), "v"(l16), "s"(rd), "s"(hd[k]) : "memory");
      wait_vm<0>();
    }
    const v4i rf = make_rsrc(a_d, 1u << 20);
    const unsigned cnt = PH;
    asm volatile("s_nop 4\n\tbuffer_atomic_add %0, %1, %2, %3 offen sc1" :: "v"(cnt), "v"(l0off), "s"(rf), "s"(OFF_IN + rank * 128 + 21 * 4) : "memory");
  }
  const unsigned tag = call & 0xffffffu;
  const unsigned tbase = (call - 1) * NPC - 3 * NPL;
  const unsigned long long a_t = a_d + OFF_OST + (call & 1) * OST_BYTES + rank * (4u << 20);
  bool dead = false, open = false;
  unsigned spins = 63;
  v4u D[2][2], E[2][2];
#pragma unroll
  for (int i = 0; i < 2; ++i) { D[i][0] = D[i][1] = E[i][0] = E[i][1] = (v4u){0, 0, 0, 0}; }
  unsigned SN[2] = {0, 0}, TK[2] = {0, 0};
  unsigned f0 = idx, f1 = idx + NPL, f2 = idx + 2 * NPL, fcnt = 3;
  unsigned req1 = 0, req2 = 0;
  unsigned so1 = NONE, do0 = NONE, do1 = NONE;
  unsigned long long tw = 0;
  auto iter = [&](auto ii) {
    constexpr int c = decltype(ii)::value, n = c ^ 1;
    {
      const v4i rs = make_rsrc(a_t, so1 == NONE ? 0u : 4u << 20);
      const unsigned so = rfl(so1 == NONE ? 0u : so1);
      asm volatile("s_nop 4\n\tbuffer_load_dwordx4 %0, %1, %2, %3 offen sc0 sc1" : "=v"(D[n][0]) : "v"(l16), "s"(rs), "s"(so) : "memory");
      asm volatile("buffer_load_dwordx4 %0, %1, %2, %3 offen offset:1024 sc0 sc1" : "=v"(D[n][1]) : "v"(l16), "s"(rs), "s"(so) : "memory");
    }
    {
      v4u o[2];
#pragma unroll
      for (int h = 0; h < 2; ++h)
#pragma unroll
        for (int j = 0; j < 4; ++j)
          o[h][j] = __float_as_uint(silu((__uint_as_float(D[c][h][j]) + __uint_as_float(E[c][h][j])) * scale));
      const v4i rd = make_rsrc(a_o, do0 == NONE ? 0u : OUT_BYTES);
      const unsigned d0 = rfl(do0 == NONE ? 0u : do0), d1 = d0 + GN * 4u;
      asm volatile("s_nop 4\n\tbuffer_store_dwordx4 %0, %1, %2, %3 offen sc1" :: "v"(o[0]), "v"(l16), "s"(rd), "s"(d0) : "memory");
      asm volatile("buffer_store_dwordx4 %0, %1, %2, %3 offen sc1\n\ts_nop 1" :: "v"(o[1]), "v"(l16), "s"(rd), "s"(d1) : "memory");
    }
    const unsigned snap = SN[c];
    if (req2) {
      const unsigned t = rfl(TK[c]) - tbase;
      f0 = fcnt == 0 ? t : f0; f1 = fcnt == 1 ? t : f1; f2 = fcnt == 2 ? t : f2;
      ++fcnt;
    }
    unsigned so2 = NONE, do2 = NONE;
    bool take = false;
    if (f0 < NPC) {
      open = open || (__builtin_amdgcn_readlane(snap, 49) == call * 32 && __builtin_amdgcn_readlane(snap, 50) == call);
      const unsigned e = __builtin_amdgcn_readlane(snap, f0 >> 6);
      take = open && (e >> 8) == tag;
      if (take) {
        const unsigned tile = e & 31, sub = f0 & 63;
        so2 = tile * (128u << 10) + sub * 2048;
        do2 = (((tile >> 3) * 128 + sub * 2) * GN + q * NSH + (tile & 7) * 256) * 4;
        f0 = f1; f1 = f2; --fcnt; tw = 0;
      } else {
        if (tw == 0) tw = __builtin_amdgcn_s_memrealtime();
        else if (do0 == NONE && do1 == NONE) {
          if (timed_out(tmo, call, tw, spins)) { f0 = NPC; dead = true; }
          if (!open && __builtin_amdgcn_s_memrealtime() - tw < 3600) __builtin_amdgcn_s_sleep(64);
        }
      }
    }
    {
      const v4i re = make_rsrc(a_o, take ? OUT_BYTES : 0u);
      const unsigned e0 = rfl(take ? do2 : 0u), e1 = e0 + GN * 4u;
      asm volatile("s_nop 4\n\tbuffer_load_dwordx4 %0, %1, %2, %3 offen sc1" : "=v"(E[c][0]) : "v"(l16), "s"(re), "s"(e0) : "memory");
      asm volatile("buffer_load_dwordx4 %0, %1, %2, %3 offen sc1" : "=v"(E[c][1]) : "v"(l16), "s"(re), "s"(e1) : "memory");
    }
    asm volatile("buffer_load_dword %0, %1, %2, %3 offen sc0 sc1" : "=v"(SN[c]) : "v"(l4), "s"(rs_my), "s"(snap_off) : "memory");
    {
      const v4i rt = make_rsrc(a_my, take ? 1u << 20 : 0u);
      TK[c] = one;
      asm volatile("s_nop 4\n\tbuffer_atomic_add %0, %1, %2, %3 offen sc0" : "+v"(TK[c]) : "v"(l0off), "s"(rt), "s"(tick_off) : "memory");
    }
    wait_vm<6>();
    asm volatile("" : "+v"(D[n][0]), "+v"(D[n][1]), "+v"(E[n][0]), "+v"(E[n][1]), "+v"(SN[n]), "+v"(TK[n]));
    req2 = req1; req1 = take;
    do0 = do1; do1 = do2; so1 = so2;
    return f0 >= NPC && fcnt != 0 && do0 == NONE && do1 == NONE;
  };
  for (;;) {
    if (iter(std::integral_constant<int, 0>{})) break;
    if (iter(std::integral_constant<int, 1>{})) break;
  }
  wait_vm<0>();
  if (dead && lane == 0) p.out[rank * NSH] = __builtin_nanf("");
}

__device__ __forceinline__ void mover(const Params& p, const int b, const int wave, const int lane) {
  const int rank = (int)rfl((unsigned)p.rank);
  const int gw = wave * NSEND + b;
  gu8* const myws = rfl_ptr(p.ws[rank]);
  if (gw < 7 * NWL) {
    pusher(p, myws, rank, gw, lane);
  } else if (gw < 7 * (NWL + NPL)) {
    puller(p, myws, rank, gw, lane);
  } else if (gw == NSEND * 8 - 1) {
    unsigned c = 0;
    if (lane == 0) c = __hip_atomic_fetch_add((gu32*)(myws + OFF_CALL + gw * 16), 1u, __ATOMIC_RELAXED, __HIP_MEMORY_SCOPE_AGENT);
    monitor(myws, rank, rfl(c) + 1, lane);
  }
}

__global__ void __launch_bounds__(512, 2) fused_kernel(Params p) {
  __shared__ __attribute__((aligned(16))) uint8_t smem[LDS_BYTES];
  const int tid = threadIdx.x;
  const int lane = tid & 63;
  const int wave = __builtin_amdgcn_readfirstlane(tid >> 6);
  const int b = blockIdx.x;
  STAMP_AT(0);
  if (b < NSEND) {
    mover(p, b, wave, lane);
    return;
  }
  const int rank = (int)rfl((unsigned)p.rank);
  gu8* const myws = rfl_ptr(p.ws[rank]);
  gu32* const tmo = (gu32*)(myws + OFF_TMO);
  const int g = lane >> 4, r = lane & 15;
  volatile __attribute__((address_space(3))) unsigned* const bc =
      (volatile __attribute__((address_space(3))) unsigned*)(smem + 3 * LDS_STAGE);

  if (tid == 0) {
    const unsigned t = __hip_atomic_fetch_add((gu32*)(myws + OFF_CALL + b * 128), 1u, __ATOMIC_RELAXED, __HIP_MEMORY_SCOPE_AGENT);
    bc[0] = t + 1;
  }
  unsigned call;
  if (b >= 128) {
    const int q = (rank + 1 + (b & 7)) & 7, ch = (b - 128) >> 3;
    const int ksl = (ch >> 1) & 3, nbq = (ch >> 3) * 4 + (ch & 1) * 2 + (wave >> 2), wq = wave & 3;
    const int hi = lane >> 5, kb = (lane >> 2) & 7, cg = lane & 3;
    const uint8_t* src = p.w + (size_t)(ksl * 128 + kb * 16 + hi * 8) * GN + q * NSH + nbq * 256 + wq * 64 + cg * 16;
    v4u B[8], A[8];
#pragma unroll
    for (int k = 0; k < 8; ++k) B[k] = *(const v4u*)(src + (size_t)k * GN);
    unsigned R[16][2];
#pragma unroll
    for (int k = 0; k < 8; ++k)
#pragma unroll
      for (int e = 0; e < 2; ++e) {
        const auto sw = __builtin_amdgcn_permlane32_swap(B[k][e], B[k][2 + e], false, false);
        R[k][e] = sw[0]; R[8 + k][e] = sw[1];
      }
#pragma unroll
    for (int u = 0; u < 4; ++u)
#pragma unroll
      for (int e = 0; e < 2; ++e) {
        const unsigned a0 = R[4 * u][e], a1 = R[4 * u + 1][e], a2 = R[4 * u + 2][e], a3 = R[4 * u + 3][e];
        const unsigned t0 = __builtin_amdgcn_perm(a1, a0, 0x05010400u), t1 = __builtin_amdgcn_perm(a1, a0, 0x07030602u);
        const unsigned t2 = __builtin_amdgcn_perm(a3, a2, 0x05010400u), t3 = __builtin_amdgcn_perm(a3, a2, 0x07030602u);
        A[0 + e][u] = __builtin_amdgcn_perm(t2, t0, 0x05040100u);
        A[2 + e][u] = __builtin_amdgcn_perm(t2, t0, 0x07060302u);
        A[4 + e][u] = __builtin_amdgcn_perm(t3, t1, 0x05040100u);
        A[6 + e][u] = __builtin_amdgcn_perm(t3, t1, 0x07060302u);
      }
    __syncthreads();
    call = rfl(bc[0]);
    const unsigned s0 = 16 * ((2 * cg + hi) + 8 * (kb & 1) + 16 * (kb >> 1));
    const unsigned dbase = (q == rank ? OFF_STAGE + (call & 1) * STAGE_BYTES + rank * SRC_BYTES + SX_W
                                      : OFF_WT + q * (1u << 20)) + ch * 65536 + wave * 8192;
    const __amdgpu_buffer_rsrc_t rw = __builtin_amdgcn_make_buffer_rsrc((void*)myws, 0, WS_NEED, 0x00020000);
#pragma unroll
    for (int e = 0; e < 8; ++e)
      __builtin_amdgcn_raw_buffer_store_b128(A[e], rw, dbase + e * 1024 + ((e & 1) ? (s0 ^ 128u) : s0), 0, AUX_SC1);
    asm volatile("s_waitcnt vmcnt(0)" ::: "memory");
    __syncthreads();
    if (tid == 0) {
      if (q == rank)
        __hip_atomic_fetch_add((gu32*)(myws + OFF_IN + rank * 128) + (ch >> 1), 32u, __ATOMIC_RELAXED, __HIP_MEMORY_SCOPE_SYSTEM);
      else
        __hip_atomic_store((gu32*)(myws + OFF_SNAP + q * 256) + 32 + ch, call << 8, __ATOMIC_RELAXED, __HIP_MEMORY_SCOPE_AGENT);
    }
  } else if (b >= 96) {
    const int rg = b - 96, ksl = wave >> 1, hf = wave & 1;
    const uint8_t* src = p.x + (size_t)(rg * 128 + hf * 64 + r) * KSH + ksl * 128 + g * 16;
    v4u B[8];
#pragma unroll
    for (int e = 0; e < 8; ++e) B[e] = *(const v4u*)(src + (e >> 1) * 16 * KSH + (e & 1) * 64);
    __syncthreads();
    call = rfl(bc[0]);
    const unsigned dbase = OFF_STAGE + (call & 1) * STAGE_BYTES + rank * SRC_BYTES + xpos(rank, rg) + ksl * xstride(rank, rg)
                           + hf * 8192 + lane * 16;
    const __amdgpu_buffer_rsrc_t rw = __builtin_amdgcn_make_buffer_rsrc((void*)myws, 0, WS_NEED, 0x00020000);
#pragma unroll
    for (int e = 0; e < 8; ++e) __builtin_amdgcn_raw_buffer_store_b128(B[e], rw, dbase + e * 1024, 0, AUX_SC1);
    asm volatile("s_waitcnt vmcnt(0)" ::: "memory");
    __syncthreads();
    if (tid < 4) {
      const int st = rg & 3;
      const int cls = (rg >> 2) == rank ? 20 : (st == 0 ? 21 : 4 + 4 * st + tid);
      __hip_atomic_fetch_add((gu32*)(myws + OFF_IN + rank * 128) + cls, 8u, __ATOMIC_RELAXED, __HIP_MEMORY_SCOPE_SYSTEM);
    } else if (tid >= 64 && tid < 72) {
      __hip_atomic_fetch_add((gu32*)(myws + OFF_SNAP + (tid - 64) * 256) + 48, 1u, __ATOMIC_RELAXED, __HIP_MEMORY_SCOPE_AGENT);
    }
  } else {
    __syncthreads();
    call = rfl(bc[0]);
  }
  STAMP_AT(7);
  const int nb = b & 7, u = (b - NSEND) >> 3;
  const int wm = wave >> 2, wn = wave & 3;
  const float scale = p.sx[0] * p.sw[0];
  const gu8* const stg = myws + OFF_STAGE + (call & 1) * STAGE_BYTES;

#pragma unroll 1
  for (int pass = u >= 14 ? 0 : 3; pass < 5; ++pass) {
    const bool mini = pass < 3;
    const int item = mini ? (u - 14) + pass * (NG - 14) : u + (pass - 3) * NG;
    if (item >= (mini ? 28 : 32)) continue;
    const bool own = item >= 28;
    const int stage = own ? item - 28 : item / 7;
    const int peer = own ? rank : (rank + 1 + item % 7) & (NR - 1);
    const int rgrp = (mini ? rank : peer) * 4 + stage;

    const unsigned wsh = nb < 4 ? 0 : 4;
    const unsigned xbit = own ? 20 : 21, xsh = 4 + 4 * stage;
    const bool xone = own || stage == 0;
    gu32* const fptr = (gu32*)(myws + OFF_MASK + b * 128);
    auto segs_ready = [&](unsigned v) {
      v = rfl(v);
      const unsigned xm = xone ? (((v >> xbit) & 1u) ? 15u : 0u) : (v >> xsh);
      const unsigned m = (v >> 24) == (call & 0xffu) ? ((v >> wsh) & xm & 15u) : 0u;
      return (unsigned)__builtin_ctz(~m);
    };
    auto wait_seg = [&](unsigned seg) {
      const unsigned long long t0 = __builtin_amdgcn_s_memrealtime();
      unsigned n, spins = 63;
      for (;;) {
        n = segs_ready(__hip_atomic_load(fptr, __ATOMIC_RELAXED, __HIP_MEMORY_SCOPE_AGENT));
        if (n > seg) break;
        if (timed_out(tmo, call, t0, spins)) { n = 0x104; break; }
        __builtin_amdgcn_s_sleep(GSLEEP);
      }
      return n;
    };
    if (wave == 0) {
      unsigned n;
      if (mini) {
        const int wi = lane < 4 ? 32 + (nb >> 2) * 8 + lane * 2 + ((nb >> 1) & 1) : 48;
        const unsigned want = lane < 4 ? call << 8 : call * 32;
        gu32* f = (gu32*)(myws + OFF_SNAP + peer * 256) + wi;
        const unsigned long long t0 = __builtin_amdgcn_s_memrealtime();
        unsigned spins = 63;
        for (;;) {
          n = 4;
          if (__all(__hip_atomic_load(f, __ATOMIC_RELAXED, __HIP_MEMORY_SCOPE_AGENT) == want)) break;
          if (timed_out(tmo, call, t0, spins)) { n = 0x104; break; }
          __builtin_amdgcn_s_sleep(8);
        }
      } else {
        n = wait_seg(0);
      }
      if (lane == 0) { bc[1] = n; bc[2] = 0; }
    }
    __syncthreads();
    unsigned avail = rfl(bc[1]);
    if (!mini) STAMP_AT(1 + (pass - 3) * 3);

    v4f acc[4][4];
#pragma unroll
    for (int i = 0; i < 4; ++i)
#pragma unroll
      for (int j = 0; j < 4; ++j) acc[i][j] = (v4f){0.f, 0.f, 0.f, 0.f};

    {
      const int nsrc = mini ? 1 : (own ? 8 : 7), nks = nsrc * 4;
      const int skip = mini || own ? 8 : peer;
      const gu8* xa = stg + (mini ? rank * SRC_BYTES : 0u) + xpos(rank, rgrp) + wave * 1024 + lane * 16;
      const gu8* wb = (mini ? myws + OFF_WT + peer * (1u << 20) : stg + SX_W) + wpos(nb) + wave * 1024 + lane * 16;
      const unsigned xstr = xstride(rank, rgrp);
      int jsi = 0, jks = 0;
      auto issue = [&](int sb) {
        const unsigned so = (unsigned)(jsi + (jsi >= skip ? 1 : 0)) * SRC_BYTES;
        uint8_t* l = smem + sb * LDS_STAGE + wave * 1024;
        const gu8* a = xa + so + (unsigned)jks * xstr;
        const gu8* w = wb + so + (unsigned)jks * 131072u;
#pragma unroll
        for (int i = 0; i < 2; ++i)
          __builtin_amdgcn_global_load_lds((const __attribute__((address_space(1))) void*)(a + i * 8192),
                                           (__attribute__((address_space(3))) void*)(l + i * 8192), 16, 0, AUX_SYS);
#pragma unroll
        for (int i = 0; i < 4; ++i)
          __builtin_amdgcn_global_load_lds((const __attribute__((address_space(1))) void*)(w + i * 8192),
                                           (__attribute__((address_space(3))) void*)(l + 16384 + i * 8192), 16, 0, AUX_SYS);
        if (++jsi == nsrc) { jsi = 0; ++jks; }
      };
      const int boff = (r & 1) * 1024 + 16 * ((r >> 1) + 8 * ((g & 1) ^ (r & 1)) + 16 * (g >> 1));
      v8i af[4], bf[4];
      auto readf = [&](int sb) {
        const uint8_t* la = smem + sb * LDS_STAGE + (wm * 4) * 2048 + lane * 16;
        const uint8_t* lb = smem + sb * LDS_STAGE + 16384 + (wn * 4) * 2048 + boff;
#pragma unroll
        for (int i = 0; i < 4; ++i) {
          const v4i lo = *(const v4i*)(la + i * 2048), hi = *(const v4i*)(la + i * 2048 + 1024);
          af[i] = __builtin_shufflevector(lo, hi, 0, 1, 2, 3, 4, 5, 6, 7);
        }
#pragma unroll
        for (int j = 0; j < 4; ++j) {
          const v4i lo = *(const v4i*)(lb + j * 2048), hi = *(const v4i*)(lb + j * 2048 + 512);
          bf[j] = __builtin_shufflevector(lo, hi, 0, 1, 2, 3, 4, 5, 6, 7);
        }
      };
      issue(0);
      issue(1);
      wait_vm<6>();
      raw_barrier();
      if (wm == 1) __builtin_amdgcn_s_barrier();
      int sb = 0;
      uint8_t* const mbox = smem + 3 * LDS_STAGE + 256 + wave * 256;
      for (int ks = 0; ks < nks; ++ks) {
        readf(sb);
        if ((avail & 7) < 4) {
          if (jsi == nsrc - 2) {
            __builtin_amdgcn_global_load_lds((const __attribute__((address_space(1))) void*)fptr,
                                             (__attribute__((address_space(3))) void*)mbox, 4, 0, AUX_SC1);
          } else if (jsi == nsrc - 1) {
            avail = segs_ready(*(volatile __attribute__((address_space(3))) unsigned*)mbox);
          } else if (jsi == 0) {
            if (avail <= (unsigned)jks) avail = wait_seg(jks);
          }
        }
        if (ks + 2 < nks) { issue(sb == 0 ? 2 : sb - 1); wait_vm<6>(); } else wait_vm<0>();
        raw_barrier();
        __builtin_amdgcn_s_setprio(1);
#pragma unroll
        for (int i = 0; i < 4; ++i)
#pragma unroll
          for (int j = 0; j < 4; ++j)
            acc[i][j] = __builtin_amdgcn_mfma_scale_f32_16x16x128_f8f6f4(af[i], bf[j], acc[i][j],   0,   1,
                                                                          0, 0x7f7f7f7f, 0, 0x7f7f7f7f);
        __builtin_amdgcn_s_setprio(0);
        __builtin_amdgcn_s_barrier();
        asm volatile("" ::: "memory");
        sb = sb == 2 ? 0 : sb + 1;
      }
      if (avail >= 0x100 && lane == 0) bc[2] = 1;
      if (wm == 0) __builtin_amdgcn_s_barrier();
      raw_barrier();
    }
    const bool ok = rfl(bc[2]) == 0;
    if (!mini) STAMP_AT(2 + (pass - 3) * 3);

    {
      const int tile = stage * 8 + nb;
      const bool toout = mini || own;
      const __amdgpu_buffer_rsrc_t ro = toout
          ? __builtin_amdgcn_make_buffer_rsrc(p.out, 0, OUT_BYTES, 0x00020000)
          : __builtin_amdgcn_make_buffer_rsrc((void*)(myws + OFF_OST + (call & 1) * OST_BYTES), 0, OST_BYTES, 0x00020000);
      const unsigned ostr = toout ? (unsigned)(GN * 4) : 1024u;
      const unsigned obase = (toout ? (unsigned)((stage * 128 * GN + peer * NSH + nb * 256) * 4)
                                    : (unsigned)((peer * 32 + tile) * (128 << 10)))
                             + (unsigned)(wm * 64 + g * 4) * ostr + wn * 256 + r * 16;
      const float sc = !ok ? __builtin_nanf("") : (own ? scale : 1.f);
      if (own) {
#pragma unroll
        for (int i = 0; i < 4; ++i)
#pragma unroll
          for (int j = 0; j < 4; ++j)
#pragma unroll
            for (int reg = 0; reg < 4; ++reg) acc[i][j][reg] = silu(acc[i][j][reg] * sc);
      } else {
#pragma unroll
        for (int i = 0; i < 4; ++i)
#pragma unroll
          for (int j = 0; j < 4; ++j) acc[i][j] *= sc;
      }
#pragma unroll
      for (int i = 0; i < 4; ++i)
#pragma unroll
        for (int reg = 0; reg < 4; ++reg) {
          const v4f o = {acc[i][0][reg], acc[i][1][reg], acc[i][2][reg], acc[i][3][reg]};
          __builtin_amdgcn_raw_buffer_store_b128(__builtin_bit_cast(v4u, o), ro, obase + (unsigned)(i * 16 + reg) * ostr, 0, AUX_SYS);
        }
      asm volatile("s_waitcnt vmcnt(0)" ::: "memory");
      __syncthreads();
      if (tid == 0) {
        if (mini) {
          __hip_atomic_fetch_add((gu32*)(myws + OFF_SNAP + peer * 256) + 49, 1u, __ATOMIC_RELAXED, __HIP_MEMORY_SCOPE_AGENT);
        } else if (!own) {
          const unsigned slot = __hip_atomic_fetch_add((gu32*)(myws + OFF_RCNT + peer * 128), 1u, __ATOMIC_RELAXED,
                                                       __HIP_MEMORY_SCOPE_AGENT) & 31u;
          __hip_atomic_store((gu32*)(p.ws[peer] + OFF_SNAP + rank * 256) + slot, (call << 8) | (unsigned)tile, __ATOMIC_RELAXED,
                             __HIP_MEMORY_SCOPE_SYSTEM);
        }
      }
    }
    if (!mini) STAMP_AT(3 + (pass - 3) * 3);
  }
}

}

extern "C" void kernel_launch(int rank, int nranks, ncclComm_t comm,
                              hipStream_t stream,
                              void* const* d_in, const int* in_sizes, int n_in,
                              void* d_out, int out_size,
                              const int* in_dtypes, int out_dtype,
                              void* const* d_out_all,
                              void* const* d_ws_all, size_t ws_bytes) {
  (void)comm; (void)in_dtypes; (void)out_dtype; (void)d_out_all;
  if (nranks != NR || n_in != 4 || ws_bytes < (size_t)WS_NEED || out_size != 512 * GN ||
      in_sizes[0] != 4096 * KSH || in_sizes[1] != KSH * GN || (unsigned)rank >= (unsigned)NR) {
    fprintf(stderr, "kernel_launch: unexpected problem configuration\n");
    return;
  }
  Params p;
  p.x = (const uint8_t*)d_in[0];
  p.w = (const uint8_t*)d_in[1];
  p.sx = (const float*)d_in[2];
  p.sw = (const float*)d_in[3];
  p.out = (float*)d_out;
  for (int i = 0; i < NR; ++i) p.ws[i] = (uint8_t*)d_ws_all[i];
  p.rank = rank;
  hipLaunchKernelGGL(fused_kernel, dim3(NWG), dim3(512), 0, stream, p);
  HIP_CHECK(hipGetLastError());
}
```

```cpp
#include <hip/hip_runtime.h>
#include <rccl/rccl.h>
#include <cstdio>
#include <cstdint>
#include <type_traits>

#define HIP_CHECK(expr) do { hipError_t _e = (expr); if (_e != hipSuccess) { \
    fprintf(stderr, "HIP error %s at %s:%d\n", hipGetErrorString(_e), __FILE__, __LINE__); \
    return; } } while (0)

typedef int      v8i __attribute__((ext_vector_type(8)));
typedef int      v4i __attribute__((ext_vector_type(4)));
typedef unsigned v4u __attribute__((ext_vector_type(4)));
typedef float    v4f __attribute__((ext_vector_type(4)));

namespace {

constexpr int NR = 8;
constexpr int GN = 16384;
constexpr int KSH = 512;
constexpr int NQ = 4096;
constexpr int NWG = 256;
constexpr int NSEND = 64;
constexpr int XK = 1;
constexpr int XF = 1;
constexpr int UO = 8 + 2 * XK;
constexpr int NWL = 5 * UO / 2;
constexpr int NPL = 33;
constexpr int NPT = 26;
constexpr unsigned XSTOP = (128 + 2 * NPL);
constexpr int GATE_R = 6, GATE_X = 7;
constexpr unsigned TSL = 700u;
constexpr int WIN_C = 20;
constexpr unsigned URG = 4, URGP = 6;
constexpr int PSLEEP = 16, LSLEEP = 64, TSPIN = 2, GSPIN = 12, PLAN_P = 8;
constexpr int PRE_T0 = 30, PRE_DT = 140;
constexpr int PLAN_K = 1620;
constexpr int PLAN_T = 1500, PLAN_G = 50;
constexpr unsigned PLAN_X = 230, PLAN_S = 300;
constexpr int MINI_T0 = 800, MINI_T1 = 3600, MINI_DT = 20;

constexpr unsigned NMIN = 10, NMAX = 28, NFU = 18;
static_assert(NFU <= 192 - 6 * NMAX, "relayed units");
constexpr int H12 = 2;
constexpr unsigned LB = 4u, RS = 4u;
constexpr unsigned C_X1 = 5, C_O = 19, C_R = 20, C_RF = C_R + (192 - 6 * NMIN - NFU + 5) / 6, C_XO = C_RF + NFU, C_END = C_XO + 16;
static_assert(C_END <= 96, "class budget");
__device__ __forceinline__ unsigned wcls(unsigned u) { return u < 8 ? 1 + (u >> 1) : 4 + (u >> 2); }
__device__ __forceinline__ unsigned rcls(unsigned k) { return C_R + k - (k < RS ? LB : 0); }
__device__ __forceinline__ unsigned pcls(unsigned u) { const unsigned c = wcls(u); return c + (c >= C_R - LB ? RS : 0); }
__device__ __forceinline__ unsigned ilv(unsigned i) {
  return i >= 16u ? i : (i & ~3u) | ((i & 1) << 1) | ((i >> 1) & 1);
}
__device__ __forceinline__ unsigned plan_n(unsigned plan, unsigned f) { return (plan >> (5 * f)) & 31; }
__device__ __forceinline__ unsigned plan_r(unsigned plan) {
  unsigned r = 192;
#pragma unroll
  for (int f = 0; f < 6; ++f) r -= plan_n(plan, f);
  return r;
}
__device__ __forceinline__ unsigned plan_pos(unsigned plan, unsigned i, unsigned f) {
  unsigned pos = 0;
#pragma unroll
  for (unsigned g = 0; g < 6; ++g) {
    const unsigned n = plan_n(plan, g);
    pos += (i > n ? i - n : 0u) + (g < f && n <= i ? 1u : 0u);
  }
  return pos;
}

constexpr unsigned SX_W = 6 * 131072, SX_O = SX_W + (2u << 20), SRC_BYTES = SX_O + 262144;

constexpr unsigned OFF_TMO   = 128;
constexpr unsigned OFF_XTF   = 3072;
constexpr unsigned OFF_PLAN  = 3584;
constexpr unsigned OFF_CNT   = 4096;
constexpr unsigned CNT_P = 1024, CNT_T = 2048;
constexpr unsigned OFF_SNAP  = 12288;
constexpr unsigned OFF_GATE  = 16384;
constexpr unsigned OFF_CALL  = 32768;
constexpr unsigned OFF_MASK  = 65536;
constexpr unsigned OFF_STAGE = 1u << 20;
constexpr unsigned STAGE_BYTES = NR * SRC_BYTES;
constexpr unsigned OFF_WT = OFF_STAGE + 2 * STAGE_BYTES;
constexpr unsigned OFF_XT = OFF_WT + (8u << 20);
constexpr unsigned LOC_BYTES = 10u << 20;
constexpr unsigned PACKED = 2 * STAGE_BYTES + 2 * LOC_BYTES;
constexpr unsigned OFF_OST = OFF_STAGE + 2 * PACKED;
constexpr unsigned OST_BYTES = 32u << 20;
constexpr unsigned WS_NEED = OFF_OST + 2 * OST_BYTES;
constexpr unsigned OUT_BYTES = 512u * GN * 4u;

constexpr unsigned long long DEADLINE_TICKS = 50000000ull;

constexpr int AUX_SC1 = 16;
constexpr int AUX_SYS = 17;
constexpr int LDS_STAGE = 48 * 1024;
constexpr int LDS_MB = 3 * LDS_STAGE + 256;
constexpr int LDS_BYTES = LDS_MB + 1280;

struct Params {
  const uint8_t* x;
  const uint8_t* w;
  const float* sx;
  const float* sw;
  float* out;
  uint8_t* ws[NR];
  int rank;
};

typedef __attribute__((address_space(1))) uint8_t gu8;
typedef __attribute__((address_space(1))) unsigned gu32;
typedef __attribute__((address_space(3))) uint8_t lu8;
typedef __attribute__((address_space(3))) v4i lv4i;
typedef __attribute__((address_space(3))) v4u lv4u;

template <int N> __device__ __forceinline__ void wait_vm() { asm volatile("s_waitcnt vmcnt(%0)" :: "i"(N) : "memory"); }
__device__ __forceinline__ void raw_barrier() {
  asm volatile("" ::: "memory");
  __builtin_amdgcn_s_waitcnt(0xc07f);
  __builtin_amdgcn_s_barrier();
  asm volatile("" ::: "memory");
}
__device__ __forceinline__ unsigned rfl(unsigned v) { return __builtin_amdgcn_readfirstlane(v); }
__device__ __forceinline__ gu8* rfl_ptr(const void* ptr) {
  const unsigned long long a = (unsigned long long)ptr;
  return (gu8*)(((unsigned long long)rfl((unsigned)(a >> 32)) << 32) | rfl((unsigned)a));
}
__device__ __forceinline__ v4i make_rsrc(unsigned long long a, unsigned bytes) {
  v4i r;
  r[0] = (int)rfl((unsigned)a);
  r[1] = (int)(rfl((unsigned)(a >> 32)) & 0xffffu);
  r[2] = (int)rfl(bytes);
  r[3] = 0x00020000;
  return r;
}
__device__ __forceinline__ bool timed_out(gu32* tmo, unsigned call, unsigned long long t0, unsigned& spins) {
  if (((++spins & 63u) == 0 && rfl(__hip_atomic_load(tmo, __ATOMIC_RELAXED, __HIP_MEMORY_SCOPE_AGENT)) == call) ||
      __builtin_amdgcn_s_memrealtime() - t0 > DEADLINE_TICKS) {
    __hip_atomic_store(tmo, call, __ATOMIC_RELAXED, __HIP_MEMORY_SCOPE_AGENT);
    return true;
  }
  return false;
}
__device__ __forceinline__ float silu(float y) {
  const float yc = __builtin_amdgcn_fmed3f(y, -60.f, 60.f);
  const float e = __builtin_amdgcn_exp2f(yc * -1.4426950408889634f);
  return y * __builtin_amdgcn_rcpf(1.f + e);
}

__device__ __forceinline__ unsigned transpose_strip(const uint8_t* w, lu8* l, const int lane) {
  v4u A[8];
  const int hi = lane >> 5, kb = (lane >> 2) & 7, cg = lane & 3;
  const uint8_t* src = w + (size_t)(kb * 16 + hi * 8) * GN + cg * 16;
  v4u B[8];
#pragma unroll
  for (int k = 0; k < 8; ++k) B[k] = *(const v4u*)(src + (size_t)k * GN);
  unsigned R[16][2];
#pragma unroll
  for (int k = 0; k < 8; ++k)
#pragma unroll
    for (int e = 0; e < 2; ++e) {
      const auto sw = __builtin_amdgcn_permlane32_swap(B[k][e], B[k][2 + e], false, false);
      R[k][e] = sw[0]; R[8 + k][e] = sw[1];
    }
#pragma unroll
  for (int u = 0; u < 4; ++u)
#pragma unroll
    for (int e = 0; e < 2; ++e) {
      const unsigned a0 = R[4 * u][e], a1 = R[4 * u + 1][e], a2 = R[4 * u + 2][e], a3 = R[4 * u + 3][e];
      const unsigned t0 = __builtin_amdgcn_perm(a1, a0, 0x05010400u), t1 = __builtin_amdgcn_perm(a1, a0, 0x07030602u);
      const unsigned t2 = __builtin_amdgcn_perm(a3, a2, 0x05010400u), t3 = __builtin_amdgcn_perm(a3, a2, 0x07030602u);
      A[0 + e][u] = __builtin_amdgcn_perm(t2, t0, 0x05040100u);
      A[2 + e][u] = __builtin_amdgcn_perm(t2, t0, 0x07060302u);
      A[4 + e][u] = __builtin_amdgcn_perm(t3, t1, 0x05040100u);
      A[6 + e][u] = __builtin_amdgcn_perm(t3, t1, 0x07060302u);
    }
  const unsigned s0 = 16 * ((2 * cg + hi) + 8 * (kb & 1) + 16 * (kb >> 1));
#pragma unroll
  for (int e = 0; e < 8; ++e) *(lv4u*)(l + e * 1024 + ((e & 1) ? (s0 ^ 128u) : s0)) = A[e];
  unsigned sum = 0;
#pragma unroll
  for (int e = 0; e < 8; ++e) sum = __builtin_amdgcn_sad_u8(A[e][e & 3] & 0x7f7f7f7fu, 0, sum);
  sum += __builtin_amdgcn_update_dpp(0u, sum, 0x111, 0xf, 0xf, true);
  sum += __builtin_amdgcn_update_dpp(0u, sum, 0x112, 0xf, 0xf, true);
  sum += __builtin_amdgcn_update_dpp(0u, sum, 0x114, 0xf, 0xf, true);
  sum += __builtin_amdgcn_update_dpp(0u, sum, 0x118, 0xf, 0xf, true);
  const int tot = __builtin_amdgcn_readlane(sum, 15) + __builtin_amdgcn_readlane(sum, 31) + __builtin_amdgcn_readlane(sum, 47) +
                  __builtin_amdgcn_readlane(sum, 63);
  return min(max(((tot + 512) >> 11) - WIN_C, 0), 96);
}

__device__ __forceinline__ void pack_piece(v4u& a0, v4u& a1, const unsigned m0, const unsigned ex, const unsigned pos, unsigned& cnt,
                                           const int lane) {
  const unsigned M0 = m0 * 0x01010101u;
  const unsigned lo60 = lane < 32 ? 0x60606060u : 0u, hi40 = lane < 32 ? 0u : 0x40404040u, hi20 = hi40 >> 1;
  unsigned y[2];
#pragma unroll
  for (int h = 0; h < 2; ++h) {
    v4u& a = h ? a1 : a0;
    unsigned x[4];
#pragma unroll
    for (int k = 0; k < 4; ++k) {
      const unsigned t = ((a[k] | 0x80808080u) - M0) & 0x1f1f1f1fu;
      x[k] = ((t + M0) ^ a[k]) & 0x7f7f7f7fu;
      a[k] = t | (a[k] & 0x80808080u);
    }
    y[h] = (x[0] >> 5) | (x[1] >> 3) | (x[2] >> 1) | (x[3] << 1);
  }
  do {
#pragma unroll
    for (int h = 0; h < 2; ++h) {
      const unsigned long long m = __ballot(y[h] != 0);
      const unsigned z = __builtin_ctz(y[h] | 0x80000000u) & 30;
      const unsigned slot = __builtin_amdgcn_mbcnt_hi((unsigned)(m >> 32), __builtin_amdgcn_mbcnt_lo((unsigned)m, cnt));
      const unsigned e = (pos + h * 1024 + lane * 16 + ((z & 6) << 1) + (z >> 3)) | (((y[h] >> z) & 3) << 12);
      const unsigned ad = ex + 2 * (y[h] != 0 && slot < 128 ? slot : 128 + lane);
      asm volatile("ds_write_b16 %0, %1" :: "v"(ad), "v"(e) : "memory");
      y[h] &= ~(3u << z);
      cnt += __builtin_popcountll(m);
    }
  } while (cnt <= 128 && __any((y[0] | y[1]) != 0));
#pragma unroll
  for (int k = 0; k < 4; ++k) {
    const unsigned u = __builtin_amdgcn_permlane32_swap(a1[k], a1[k], false, false)[1];
    a0[k] |= ((u << 5) & lo60) | ((u >> 1) & hi40) | ((u << 1) & hi20);
    a1[k] |= (u << 3) & 0x60606060u;
  }
}

__device__ __forceinline__ void planner(gu8* myws, unsigned call, int lane) {
  gu32* const tmo = (gu32*)(myws + OFF_TMO);
  const unsigned long long t0 = __builtin_amdgcn_s_memrealtime();
  unsigned spins = 0, pa = 0;
  gu32* const f = (gu32*)(lane < 7 ? myws + OFF_CNT + (call & 1) * 4096 + CNT_T + lane * 128 : myws + OFF_MASK + 255 * 128);
  int early = 0;
  while (__builtin_amdgcn_s_memrealtime() - t0 < PLAN_S) __builtin_amdgcn_s_sleep(16);
  for (;;) {
    unsigned v = 0;
    if (lane < 8) v = __hip_atomic_load(f, __ATOMIC_RELAXED, __HIP_MEMORY_SCOPE_AGENT);
    unsigned wa = __builtin_amdgcn_readlane(v, 7);
    wa = (wa >> 30) == (call & 3) ? wa : 0u;
    int nn = 0;
#pragma unroll
    for (int g = 0; g < 6; ++g) nn += plan_n(wa, g) != 0 && plan_n(pa, g) == 0;
    early += nn * max(PLAN_T - (int)(__builtin_amdgcn_s_memrealtime() - t0), 0);
    pa = wa;
    unsigned k[7], mx = 0, sum = 0;
#pragma unroll
    for (int l = 0; l < 7; ++l) { k[l] = __builtin_amdgcn_readlane(v, l); mx = max(mx, k[l]); sum += k[l]; }
    if (mx >= PLAN_X) {
      unsigned plan = call << 30;
#pragma unroll
      for (int l = 1; l < 7; ++l) {
        const int n = min(max((PLAN_K + early * PLAN_G / 600 + (int)(7 * k[l] - sum) + 56) / 112, (int)NMIN), (int)NMAX);
        plan |= (unsigned)n << (5 * (l - 1));
      }
      if (lane == 0) __hip_atomic_store((gu32*)(myws + OFF_PLAN), plan, __ATOMIC_RELAXED, __HIP_MEMORY_SCOPE_AGENT);
      if (lane < 8)
        __hip_atomic_store((__attribute__((address_space(1))) unsigned long long*)(myws + OFF_SNAP + lane * 256 + 56 * 4),
                           ((unsigned long long)call << 32) | plan, __ATOMIC_RELAXED, __HIP_MEMORY_SCOPE_AGENT);
      return;
    }
    if (timed_out(tmo, call, t0, spins)) return;
    __builtin_amdgcn_s_sleep(PLAN_P);
  }
}

__device__ __forceinline__ void monitor(gu8* myws, gu8* pws, int rank, unsigned call, int lane) {
  gu32* const tmo = (gu32*)(myws + OFF_TMO);
  const unsigned long long t0 = __builtin_amdgcn_s_memrealtime();
  unsigned spins = 0;
  gu8* const cnt = myws + OFF_CNT + (call & 1) * 4096;
  {
    const __amdgpu_buffer_rsrc_t rz = __builtin_amdgcn_make_buffer_rsrc((void*)myws, 0, 1u << 20, 0x00020000);
#pragma unroll
    for (int i = 0; i < 3; ++i)
      __builtin_amdgcn_raw_buffer_store_b128((v4u){0, 0, 0, 0}, rz, OFF_CNT + (~call & 1) * 4096 + i * 1024 + lane * 16, 0, AUX_SYS);
  }
  const unsigned lf = min(lane / 5, 5), lg = lane < 30 ? lane % 5 : lane - 32;
  const gu8* f = lane == 63 ? myws + OFF_PLAN : cnt + (lane < 30 ? lf * 128 + lg * 16 : lane < 32 || lane >= 56 ? 0 : CNT_P + lg * 16);
  unsigned tgt[4];
  auto targets = [&](unsigned lw, unsigned rc) {
#pragma unroll
    for (int k = 0; k < 4; ++k) {
      const unsigned c = lg * 4 + k;
      tgt[k] = lane < 30 ? (c == 0 ? 20u * UO + 40 * XF : c < C_X1 ? 32u : c == C_X1 ? 40u * (4 - XK - XF) : c == C_O ? 128u : min(64u, lw - min(lw, (c - 4) * 64)))
               : lane < 32 || lane >= 56 ? 0u
               : c == 0 ? 24u * UO + 48 * XF : c < C_X1 ? 32u : c == C_X1 ? 48u * (4 - XK - XF) : c < C_R - LB || (c >= C_R - LB + RS && c < C_R + RS) ? 64u
               : c < C_RF ? min(96u, rc - min(rc, (c - C_R + (c < C_R - LB + RS ? LB : 0)) * 96)) : c < C_XO ? 16u : c < C_END ? 8u : 0u;
    }
  };
  targets(4096, 4096);
  const unsigned tag = call << 30;
  unsigned pa = 0, pb = 0, plan = 0;
  bool told = false, pub = false;
  v4u nx;
  asm volatile("global_load_dwordx4 %0, %1, off sc0 sc1" : "=v"(nx) : "v"(f) : "memory");
  for (;;) {
    v4u v;
    if (!pub) wait_vm<0>();
    asm volatile("s_waitcnt vmcnt(3)\n\tv_mov_b32 %0, %4\n\tv_mov_b32 %1, %5\n\tv_mov_b32 %2, %6\n\tv_mov_b32 %3, %7"
                 : "=&v"(v[0]), "=&v"(v[1]), "=&v"(v[2]), "=&v"(v[3]) : "v"(nx[0]), "v"(nx[1]), "v"(nx[2]), "v"(nx[3]) : "memory");
    asm volatile("global_load_dwordx4 %0, %1, off sc0 sc1" : "+v"(nx) : "v"(f) : "memory");
    bool news = false;
    if (plan == 0) {
      const unsigned pl = __builtin_amdgcn_readlane(v[0], 63);
      if ((pl >> 30) == (call & 3) && (pl & 31) >= NMIN) {
        plan = pl;
        targets(512 + 16 * plan_n(plan, lf), (plan_r(plan) - NFU) * 16);
        news = true;
      }
    }
    unsigned lm = 4;
#pragma unroll
    for (int k = 3; k >= 0; --k) lm = v[k] < tgt[k] ? k : lm;
    const unsigned long long miss = __ballot(lm < 4);
    unsigned wa = tag, wmin = 20;
#pragma unroll
    for (int g = 0; g < 6; ++g) {
      const unsigned m = ((unsigned)miss >> (5 * g)) & 31, z = __builtin_ctz(m | 16);
      const unsigned pf = m ? 4 * z + __builtin_amdgcn_readlane(lm, 5 * g + z) : 20u;
      wa |= pf << (5 * g);
      wmin = min(wmin, pf);
    }
    const unsigned hi = (unsigned)(miss >> 32), hz = __builtin_ctz(hi | 0x80000000u);
    const unsigned wb = (call << 16) | (hi ? 4 * hz + __builtin_amdgcn_readlane(lm, 32 + hz) : 128u);
    pub = wa != pa || wb != pb || news;
    if (pub) {
      const v4u o = {wa, wb, plan, 0};
#pragma unroll
      for (int i = 1; i < 4; ++i)
        asm volatile("global_store_dwordx4 %0, %1, off sc1\n\ts_nop 1" :: "v"(myws + OFF_MASK + (i * 64 + lane) * 128), "v"(o) : "memory");
      if (plan != 0 && !told && (wa != pa || news) && lane == 0)
        __hip_atomic_store((__attribute__((address_space(1))) unsigned long long*)(pws + OFF_SNAP + rank * 256 + 58 * 4),
                           ((unsigned long long)plan << 32) | wa, __ATOMIC_RELAXED, __HIP_MEMORY_SCOPE_SYSTEM);
      pa = wa; pb = wb;
      if (wmin >= C_O && !told) {
        told = true;
        if (lane == 0) __hip_atomic_store((gu32*)(pws + OFF_SNAP + rank * 256) + 52, call, __ATOMIC_RELAXED, __HIP_MEMORY_SCOPE_SYSTEM);
      }
    }
    if (miss == 0) break;
    if (timed_out(tmo, call, t0, spins)) break;
  }
  wait_vm<0>();
}

__device__ __forceinline__ unsigned push_start(const Params& p, const int gw, const int lane, v4u (&X)[4][2], unsigned& xo) {
  const int link = gw % 7, idx = gw / 7;
  int rank = p.rank;
  unsigned long long a_d = (unsigned long long)p.ws[link + 1], a_my = (unsigned long long)p.ws[0], a_x = (unsigned long long)p.x;
  asm volatile("" : "+s"(rank), "+s"(a_d), "+s"(a_my), "+s"(a_x));
  const int q = rank ^ (link + 1);
  const unsigned x_voff = (unsigned)((lane & 15) * KSH + (lane >> 4) * 16), l16 = lane * 16;
  const unsigned l0off = lane == 0 ? 0u : 0x40000000u;
  const v4i rs_my = make_rsrc(a_my, 1u << 20);
  const v4i rs_x = make_rsrc(a_x, 4096u * KSH);
  unsigned so[2], dso[2];
  auto unit = [&](int j) {
    const unsigned un = idx + j * NWL;
    const unsigned z = un / UO, fo = z + (link != 0 && z + 1 >= (unsigned)link ? 1u : 0u), y = un % UO;
    const unsigned rg = (q ^ (fo + 2)) * 4 + (q & 1) * 2;
    so[j] = rfl(((rg + (y >> 3)) * 128 + (y & 1) * 64) * KSH + ((y >> 1) & 3) * 128);
    dso[j] = rfl(fo * 131072 + y * 8192);
  };
  unsigned callv = 1;
  asm volatile("s_nop 4\n\tbuffer_atomic_add %0, %1, %2, %3 offen sc0" : "+v"(callv) : "v"(l0off), "s"(rs_my), "s"(OFF_CALL + gw * 16) : "memory");
  v4u D[4][2];
#define LOAD_X(R, u) _Pragma("unroll") for (int j = 0; j < 4; ++j) { \
    const unsigned o = so[u] + j * (16 * KSH); \
    asm volatile("s_nop 4\n\tbuffer_load_dwordx4 %0, %2, %3, %4 offen\n\tbuffer_load_dwordx4 %1, %2, %3, %4 offen offset:64" \
                 : "=&v"(R[j][0]), "=&v"(R[j][1]) : "v"(x_voff), "s"(rs_x), "s"(o) : "memory"); }
  unit(0);
  LOAD_X(D, 0)
  unit(1);
  wait_vm<4>();
  asm volatile("" : "+v"(callv), "+v"(D[0][0]), "+v"(D[0][1]), "+v"(D[1][0]), "+v"(D[1][1]));
  const unsigned call = rfl(callv) + 1;
  const unsigned base = OFF_STAGE + (call & 1) * STAGE_BYTES + rank * SRC_BYTES;
  const v4i rd = make_rsrc(a_d, WS_NEED);
#pragma unroll
  for (int k = 0; k < 4; ++k) {
    const unsigned d0 = base + dso[0] + k * 2048;
    asm volatile("s_nop 4\n\tbuffer_store_dwordx4 %0, %2, %3, %4 offen sc0 sc1\n\t"
                 "buffer_store_dwordx4 %1, %2, %3, %4 offen offset:1024 sc0 sc1\n\ts_nop 1"
                 :: "v"(D[k][0]), "v"(D[k][1]), "v"(l16), "s"(rd), "s"(d0) : "memory");
    if (k == 1) {
      LOAD_X(X, 1)
      wait_vm<8>();
      asm volatile("" : "+v"(D[2][0]), "+v"(D[2][1]), "+v"(D[3][0]), "+v"(D[3][1]));
    }
  }
  wait_vm<4>();
#pragma unroll
  for (int j = 0; j < 4; ++j) asm volatile("" : "+v"(X[j][0]), "+v"(X[j][1]));
  xo = base + dso[1];
  return call;
}

template <bool PAIR>
__device__ __forceinline__ void puller(const Params& p, gu8* myws, const int rank, const int gw, const int lane, const unsigned lds,
                                       const unsigned call, const v4u (&X)[4][2], const unsigned xo) {
  constexpr unsigned NONE = 0xff, TILE = 0xfe;
  constexpr unsigned OOB = 0x80000000u, NOSV = ~0u, BIG = 1u << 21;
  const unsigned long long tstart = __builtin_amdgcn_s_memrealtime();
  gu32* const tmo = (gu32*)(myws + OFF_TMO);
  const int link = gw % 7, idx = gw / 7;
  const int q = rank ^ (link + 1);
  const v4i rs_my = make_rsrc((unsigned long long)myws, WS_NEED), rs_o = make_rsrc((unsigned long long)p.out, OUT_BYTES);
  const v4i rs_d = make_rsrc((unsigned long long)p.ws[link + 1], WS_NEED);
  const unsigned l16 = lane * 16, l4 = lane * 4;
  const unsigned l0off = lane == 0 ? 0u : 0x40000000u, l40 = lane < 40 ? l16 : 0x40000000u;
  const unsigned snap_off = OFF_SNAP + q * 256;
  float scale = PAIR ? 0.f : p.sx[0] * p.sw[0];
  constexpr unsigned no = PAIR ? 6 : 5, nx0 = PAIR ? 4 * UO : 0, nx = nx0 + 8 * no * XF, nx1 = 8 * no * (4 - XK - XF);
  constexpr unsigned nfree = nx + nx1 + (PAIR ? 1024 - 64 * LB : 32 * NMIN);
  unsigned nin = 1u << 20, npc = 1u << 20, nd = 0;
  unsigned tstop = BIG;
  const unsigned tick_off = OFF_CNT + (call & 1) * 4096 + CNT_T + link * 128;
  const unsigned o_tiles = OFF_OST + (call & 1) * OST_BYTES + rank * (4u << 20);
  const unsigned o_stage = OFF_STAGE + (call & 1) * STAGE_BYTES;
  const unsigned obase = ((q & 1) * 256 * GN + (q >> 1) * NQ) * 4;
  const unsigned flag_off = OFF_CNT + (call & 1) * 4096 + (PAIR ? CNT_P : (link - 1) * 128);
  const unsigned o_loc = (call & 1) * LOC_BYTES;
  const unsigned o_w = OFF_WT + o_loc + (rank >> 1) * (2u << 20), o_q = o_stage + q * SRC_BYTES + SX_W;
  const unsigned hi80 = lane < 32 ? 0u : 0x80808080u, hi10 = hi80 >> 3;
  const unsigned la16 = lds + l16, la2 = lds + lane * 2;
  const unsigned thr = call * (lane == 48 ? (PAIR ? 11 : 9) : lane == 52 || lane == 57 ? 1 : 8);
  constexpr unsigned long long RVALID = 0xf211ff00ffffffffull;
  auto entry = [&](unsigned t, unsigned so, unsigned dn) __attribute__((always_inline)) {
    asm volatile("ds_write_b64 %0, %1 offset:8192" :: "v"(lds + t), "v"(((unsigned long long)dn << 32) | so) : "memory");
  };
  auto table = [&](unsigned plan) __attribute__((always_inline)) {
#pragma unroll 1
    for (unsigned j = lane; j < 192; j += 64) {
      if (PAIR && nd != 0) {
        const unsigned i = j / 6, g = j % 6, pos = plan_pos(plan, i, g);
        const unsigned c = pos + NFU >= nd ? C_RF + pos + NFU - nd : rcls(pos / 6);
        const unsigned t = nfree + pos * 16 + (pos >= min(6 * RS, nd - NFU) ? 64 * LB : 0);
        const unsigned dn = o_stage + (rank ^ (2 + g)) * SRC_BYTES + SX_W + (ilv(i) * 2 + (q & 1)) * 32768;
        if (i >= plan_n(plan, g)) {
          entry(t, dn + PACKED + i * 8 + g, dn | (1 << 13) | (52 << 7) | c);
          entry(t + 8, dn + 16384 + PACKED + i * 8 + g, (dn + 16384) | (1 << 13) | (52 << 7) | c);
        }
      } else {
        unsigned y = j * 8;
        const unsigned c = wcls(y >> 4), t = y + nx + (y < 128 ? 0 : nx1);
        if (!PAIR) {
          if (y >= 32 * nd && nd != 0) y = ((nd + ((y - 32 * nd) >> 4)) * 2 + (rank & 1)) * 16 + (y & 15);
          y = ilv(y >> 5) * 32 + (y & 31);
        }
        const unsigned rb = t >= nfree && nd == 0 ? 33 : 40 + ((y >> 7) & 7);
        y = (y & 1023) * 2048;
        entry(t, (PAIR ? o_q : o_w) + PACKED + y, (o_q + y) | (1 << 13) | (rb << 7) | c);
      }
    }
    if (nd != 0) {
      if (PAIR && lane < 8 * LB) {
        const unsigned y = 1024 - 64 * LB + lane * 8;
        entry(nfree + min(6 * RS, nd - NFU) * 16 + lane * 8, o_q + PACKED + y * 2048, (o_q + y * 2048) | (1 << 13) | ((40 + (y >> 7)) << 7) | pcls(y >> 4));
      }
      if (lane < 16)
        entry(nin - 128 + lane * 8, OFF_XT + o_loc + rank * 262144 + lane * 16384,
              (o_stage + q * SRC_BYTES + SX_O + lane * 16384) | (48 << 7) | (PAIR ? C_XO + lane : C_O));
    } else if (lane < (int)(nx + nx1) / 8) {
      const bool f0 = lane < (int)nx0 / 8, f = lane < (int)nx / 8;
      const unsigned k = lane - nx0 / 8, z = k % no, fo = f0 ? 5 : z + (!PAIR && z + 1 >= (unsigned)link ? 1u : 0u);
      const unsigned y = (f0 ? lane : 4 + XK + k / no) * 16384;
      entry(lane * 8 + (f ? 0 : 128), OFF_XT + o_loc + ((rank ^ (fo + 2)) * 4 + (rank & 1) * 2) * 65536 + y,
            (o_stage + q * SRC_BYTES + fo * 131072 + y) | (48 << 7) | (f ? 0 : C_X1));
    }
  };
  bool dead = false;
  unsigned spins = 63;
  v4u D[2][2], E[2][2];
#pragma unroll
  for (int i = 0; i < 2; ++i) { D[i][0] = D[i][1] = E[i][0] = E[i][1] = (v4u){0, 0, 0, 0}; }
  unsigned SN[2], TK[2] = {0, 0}, sv[2];
  const bool push = idx < NWL;
  sv[0] = BIG; sv[1] = NOSV;
  asm volatile("ds_write_b128 %0, %1\n\tds_write_b128 %0, %1 offset:1024\n\tds_write_b128 %0, %1 offset:2048\n\tds_write_b128 %0, %1 offset:3072"
               :: "v"(la16), "v"(D[0][0]) : "memory");
  table(0);
  unsigned do0 = OOB, do1 = OOB, c0 = NONE, c1 = NONE, cp = NONE;
  unsigned acls = 0, acnt = 0;
  unsigned long long tw = 0;
  unsigned so1 = OOB, hdr = 0, xa1 = lds, xv1 = 0, tbase = 0;
  auto isw = [&](unsigned c) { return c - 1 < (PAIR ? C_XO : C_O) - 1 && c != C_X1; };
  auto iter = [&](auto ii) {
    constexpr int c = decltype(ii)::value, n = c ^ 1;
    unsigned t = sv[c] != NOSV ? sv[c] : rfl(TK[c]) + tbase;
    const unsigned snap = SN[c];
    const unsigned long long rm = __ballot((int)(snap - thr) >= 0) & RVALID;
    if (c == 1 && __builtin_expect(nd == 0, 0) && ((rm >> 57) & 1)) {
      const unsigned plan = __builtin_amdgcn_readlane(snap, 56);
      nd = PAIR ? plan_r(plan) : plan_n(plan, link - 1);
      nin = (PAIR ? 1024 : 512) + nx + nx1 + 16 * nd + 128;
      npc = nin;
      if (idx >= NPT) tstop = nin - XSTOP;
      table(plan);
    }
    if (!PAIR && c == 1 && __builtin_expect(t >= npc, 0) && tbase == 0 && idx < NPT && !dead) {
      t = nin + idx; sv[0] = t + NPT; tbase = nin + 2 * NPT; npc = nin + 2048;
    }
    const unsigned d0 = rfl(do0);
    unsigned long long ent = 0;
    if (c == 1) asm volatile("ds_read_b64 %0, %1 offset:8192" : "=v"(ent) : "v"(lds + (t & 0xff8)) : "memory");
    if (isw(c0)) {
      v4u o0 = D[c][0], o1 = D[c][1];
      unsigned e;
      asm volatile("ds_write_b128 %3, %2 offset:4096\n\tds_read_u16 %0, %4 offset:4608" : "=&v"(e), "+v"(o0), "+v"(o1) : "v"(la16), "v"(la2) : "memory");
      if (c == 1) hdr = __builtin_amdgcn_readlane(o1[0], 32);
      const unsigned M0 = (hdr & 127) * 0x01010101u;
      v4u r0, r1;
#pragma unroll
      for (int k = 0; k < 4; ++k) {
        r0[k] = ((o0[k] >> 5) & 0x03030303u) | ((o1[k] >> 3) & 0x0c0c0c0cu);
        r1[k] = o1[k] & 0x9f9f9f9fu;
      }
      asm volatile("s_waitcnt lgkmcnt(0)" : "+v"(e), "+v"(r0), "+v"(r1) :: "memory");
      const unsigned xa = lds + (e & 0xffc), xv = ((e >> 7) & 0x60) << ((e & 3) * 8);
      v4u m0, m1;
      if constexpr (c == 1) {
        asm volatile("ds_xor_b32 %3, %4\n\tds_read_b128 %0, %5\n\tds_read_b128 %1, %5 offset:1024"
                     : "=&v"(m0), "=&v"(m1), "+v"(o0) : "v"(xa), "v"(xv), "v"(la16) : "memory");
        xa1 = xa; xv1 = xv;
      } else {
        asm volatile("ds_xor_b32 %3, %4\n\tds_read_b128 %0, %5 offset:2048\n\tds_read_b128 %1, %5 offset:3072\n\t"
                     "ds_xor_b32 %3, %4\n\tds_xor_b32 %6, %7"
                     : "=&v"(m0), "=&v"(m1), "+v"(o0) : "v"(xa), "v"(xv), "v"(la16), "v"(xa1), "v"(xv1) : "memory");
      }
#pragma unroll
      for (int k = 0; k < 4; ++k) {
        const unsigned u = __builtin_amdgcn_permlane32_swap(r1[k], r0[k], false, false)[0];
        r1[k] = (u | ((o0[k] << 1) & hi80) | ((o0[k] >> 1) & hi10)) + M0;
        r0[k] = (o0[k] & 0x9f9f9f9fu) + M0;
      }
      asm volatile("s_waitcnt lgkmcnt(0)" : "+v"(m0), "+v"(m1), "+v"(r0), "+v"(r1) :: "memory");
      r0 ^= m0; r1 ^= m1;
      if (__builtin_expect(hdr & 0x4000, 0)) {
        const unsigned so = PAIR ? d0 : d0 - o_q + o_w;
        asm volatile("s_nop 4\n\tbuffer_load_dwordx4 %0, %2, %3, %4 offen sc0 sc1\n\t"
                     "buffer_load_dwordx4 %1, %2, %3, %4 offen offset:1024 sc0 sc1\n\ts_waitcnt vmcnt(0)"
                     : "=&v"(r0), "=&v"(r1) : "v"(l16), "s"(rs_d), "s"(so) : "memory");
      }
      asm volatile("s_nop 4\n\tbuffer_store_dwordx4 %0, %2, %3, %4 offen sc0 sc1\n\t"
                   "buffer_store_dwordx4 %1, %2, %3, %4 offen offset:1024 sc0 sc1\n\ts_nop 1"
                   :: "v"(r0), "v"(r1), "v"(l16), "s"(rs_my), "s"(d0) : "memory");
    } else if (!PAIR && c0 == TILE) {
      v4u o0, o1;
#pragma unroll
      for (int j = 0; j < 4; ++j) {
        o0[j] = __float_as_uint(silu((__uint_as_float(D[c][0][j]) + __uint_as_float(E[c][0][j])) * scale));
        o1[j] = __float_as_uint(silu((__uint_as_float(D[c][1][j]) + __uint_as_float(E[c][1][j])) * scale));
      }
      asm volatile("s_nop 4\n\tbuffer_store_dwordx4 %0, %2, %3, %4 offen sc0 sc1\n\t"
                   "buffer_store_dwordx4 %1, %2, %3, %5 offen sc0 sc1\n\ts_nop 1"
                   :: "v"(o0), "v"(o1), "v"(l16), "s"(rs_o), "s"(d0), "s"(d0 + GN * 4u) : "memory");
    } else {
      asm volatile("s_nop 4\n\tbuffer_store_dwordx4 %0, %2, %3, %4 offen sc0 sc1\n\t"
                   "buffer_store_dwordx4 %1, %2, %3, %4 offen offset:1024 sc0 sc1\n\ts_nop 1"
                   :: "v"(D[c][0]), "v"(D[c][1]), "v"(l16), "s"(rs_my), "s"(d0) : "memory");
    }
    if (c == 1) asm volatile("s_waitcnt lgkmcnt(0)" : "+v"(ent) :: "memory");
    const unsigned ex = rfl((unsigned)ent), ey = rfl(ent >> 32);
    unsigned so2 = (ex & ~0x3fffu) + (t & 7) * 2048, do2 = (ey & ~0x3fffu) + (t & 7) * 2048, c2 = ey & 127;
    bool take = (rm >> ((ey >> 7) & 63)) & 1;
    if (!PAIR && t >= nin) {
      const unsigned y = t - nin;
      take = (rm >> ((y >> 6) & 31)) & (rm >> (60 + ((y >> 9) & 3))) & 1;
      so2 = o_tiles + y * 2048;
      do2 = obase + ((y & 127) << 17) + ((y >> 7) << 10);
      c2 = TILE;
    }
    if (c == 0 && c1 < TILE) {
      so2 = so1 + 2048; do2 = do1 + 2048; c2 = c1; take = false;
    } else if (__builtin_expect(!take || t >= npc, 0)) {
      take = false;
      if (t < npc) {
        if (PAIR && ((ey >> 7) & 63) == 52) {
          const unsigned qa = __builtin_amdgcn_readlane(snap, 58), qp = __builtin_amdgcn_readlane(snap, 59);
          const unsigned i = (ex >> 3) & 31, f = (ex & 7) ^ 1, qn = plan_n(qp, f);
          take = (qa >> 30) == (call & 3) && (qp >> 30) == (call & 3) && qn >= NMIN &&
                 plan_n(qa, f) > wcls(i < qn ? 2 * i + (q & 1) : qn + i);
        }
        if (!PAIR && c2 == TILE && min(cp, min(c0, c1)) >= TILE) {
          gu32* const w = (gu32*)(myws + snap_off) + ((lane & 1) ? 60 + ((t - nin) >> 9) : (t - nin) >> 6);
          const unsigned long long t0 = __builtin_amdgcn_s_memrealtime();
          take = true;
          while (__any((int)(__hip_atomic_load(w, __ATOMIC_RELAXED, __HIP_MEMORY_SCOPE_SYSTEM) - call * 8) < 0)) {
            if (timed_out(tmo, call, t0, spins)) { dead = true; break; }
            __builtin_amdgcn_s_sleep(TSPIN);
          }
        } else if (!take) {
          if (tw == 0) tw = __builtin_amdgcn_s_memrealtime();
          else if (c0 == NONE && c1 == NONE) dead = timed_out(tmo, call, tw, spins);
        }
        if (dead) { take = false; npc = 0; nd = 1; sv[n] = BIG; }
      }
      if (!take) { c2 = NONE; so2 = OOB; do2 = OOB; }
    } else {
      tw = 0;
    }
    sv[c] = !take ? t : t >= tstop ? BIG : NOSV;
    if constexpr (!PAIR) {
      const unsigned e0 = c2 == TILE ? do2 : OOB;
      asm volatile("s_nop 4\n\tbuffer_load_dwordx4 %0, %2, %3, %4 offen sc1\n\tbuffer_load_dwordx4 %1, %2, %3, %5 offen sc1"
                   : "=&v"(E[c][0]), "=&v"(E[c][1]) : "v"(l16), "s"(rs_o), "s"(e0), "s"(e0 + GN * 4u) : "memory");
    }
    TK[c] = tbase == 0 ? 2 : 1;
    asm volatile("s_nop 4\n\tbuffer_load_dword %0, %2, %3, %4 offen sc0 sc1\n\tbuffer_atomic_add %1, %5, %3, %6 offen sc0"
                 : "=&v"(SN[c]), "+v"(TK[c]) : "v"(l4), "s"(rs_my), "s"(snap_off), "v"(l0off), "s"(rfl(sv[c] != NOSV ? OOB : tbase == 0 ? tick_off : tick_off + 64)) : "memory");
    {
      const bool urgent = c == 0 && (c0 <= URG || (PAIR && c0 - (C_XO - URGP) < TILE - (C_XO - URGP)) || (c0 < TILE && c1 >= TILE));
      if (urgent) wait_vm<PAIR ? 2 : 4>();
      so2 = rfl(so2);
      if constexpr (PAIR) {
        asm volatile("s_nop 4\n\ts_waitcnt vmcnt(5)\n\tbuffer_load_dwordx4 %0, %5, %6, %7 offen sc0 sc1"
                     : "=&v"(D[c][0]), "+v"(D[n][0]), "+v"(D[n][1]), "+v"(SN[n]), "+v"(TK[n]) : "v"(l16), "s"(rs_d), "s"(so2) : "memory");
      } else {
        asm volatile("s_nop 4\n\ts_waitcnt vmcnt(9)\n\tbuffer_load_dwordx4 %0, %7, %8, %9 offen sc0 sc1"
                     : "=&v"(D[c][0]), "+v"(D[n][0]), "+v"(D[n][1]), "+v"(SN[n]), "+v"(TK[n]), "+v"(E[n][0]), "+v"(E[n][1])
                     : "v"(l16), "s"(rs_d), "s"(so2) : "memory");
      }
      unsigned lb = isw(c2) ? l40 : l16;
      if (c == 0 && isw(c2)) lb |= ~(D[1][1][0] << 15) & hi80 << 7 & 0x40000000u;
      if constexpr (PAIR) {
        asm volatile("s_nop 4\n\tbuffer_load_dwordx4 %0, %1, %2, %3 offen offset:1024 sc0 sc1"
                     : "=&v"(D[c][1]) : "v"(lb), "s"(rs_d), "s"(so2) : "memory");
      } else {
        const unsigned r1 = c1 - (C_X1 + 1) < C_O - (C_X1 + 1) && ((do1 >> 15) & 1) == (unsigned)(rank & 1) ? do1 + PACKED : OOB;
        asm volatile("s_nop 4\n\tbuffer_load_dwordx4 %0, %1, %2, %3 offen offset:1024 sc0 sc1\n\t"
                     "buffer_store_dwordx4 %4, %6, %7, %8 offen sc0 sc1\n\t"
                     "buffer_store_dwordx4 %5, %9, %7, %8 offen offset:1024 sc0 sc1\n\ts_nop 1"
                     : "=&v"(D[c][1]) : "v"(lb), "s"(rs_d), "s"(so2), "v"(D[n][0]), "v"(D[n][1]), "v"(l16), "s"(rs_my), "s"(r1), "v"(l40) : "memory");
      }
#define PUBLISH(cd, cn) { acnt += cd < TILE ? 1u : 0u; acls = cd < TILE ? cd : acls; \
      const bool flush = acnt != 0 && cn != acls; \
      asm volatile("s_nop 4\n\tbuffer_atomic_add %0, %1, %2, %3 offen sc1" :: "v"(acnt), "v"(l0off), "s"(rs_my), "s"(flush ? flag_off + acls * 4 : OOB) : "memory"); \
      acnt = flush ? 0u : acnt; }
      PUBLISH(cp, c0)
      cp = c0;
      if (urgent) { PUBLISH(c0, c1) cp = NONE; }
    }
    c0 = c1; c1 = c2;
    do0 = do1; do1 = do2; so1 = so2;
    return __builtin_expect((c0 & c1) == NONE, 0) && cp >= TILE && acnt == 0 && sv[0] - npc < BIG + 1 && sv[1] - npc < BIG + 1 &&
           (PAIR || tbase != 0 || idx >= NPT || dead);
  };
  {
    const unsigned goff = OFF_GATE + (q * 32 + (push ? idx : (idx - NWL) * 2)) * 64 + (push ? 0 : 4);
    const unsigned sgo = lane == 32 ? goff : snap_off + l4, thg = lane == 32 ? call : thr;
    unsigned t0 = 2;
    SN[0] = thg - 1; TK[1] = 2;
#define TICKET(r) asm volatile("s_nop 4\n\tbuffer_atomic_add %0, %1, %2, %3 offen sc0" : "+v"(r) : "v"(l0off), "s"(rs_my), "s"(tick_off) : "memory")
    asm volatile("" : "+v"(scale));
    if (push) {
#pragma unroll
      for (int k = 0; k < 4; ++k) {
        asm volatile("s_nop 4\n\ts_waitcnt vmcnt(0)\n\tbuffer_store_dwordx4 %2, %4, %5, %6 offen sc0 sc1\n\t"
                     "buffer_store_dwordx4 %3, %4, %5, %6 offen offset:1024 sc0 sc1\n\ts_nop 1"
                     : "+v"(SN[0]), "+v"(t0) : "v"(X[k][0]), "v"(X[k][1]), "v"(l16), "s"(rs_d), "s"(xo + k * 2048) : "memory");
        if (k + 4 == GATE_R)
          asm volatile("s_nop 4\n\tbuffer_store_dword %0, %1, %2, %3 offen sc0 sc1" :: "v"(call), "v"(l0off), "s"(rs_d), "s"(OFF_GATE + (rank * 32 + idx) * 64) : "memory");
        if (k + 4 == GATE_X)
          asm volatile("s_nop 4\n\tbuffer_store_dword %0, %1, %2, %3 offen sc0 sc1" :: "v"(call), "v"(l0off), "s"(rs_d), "s"(OFF_GATE + (rank * 32 + idx) * 64 + 4) : "memory");
        if (k == 2) {
          TICKET(t0);
          asm volatile("s_nop 4\n\tbuffer_load_dword %0, %1, %2, 0 offen sc0 sc1" : "=v"(SN[0]) : "v"(sgo), "s"(rs_my) : "memory");
        }
      }
    } else {
      while (__builtin_amdgcn_s_memrealtime() - tstart < TSL) __builtin_amdgcn_s_sleep(16);
      gu32* const gate = (gu32*)(myws + goff);
      while (!dead && (int)(rfl(__hip_atomic_load(gate, __ATOMIC_RELAXED, __HIP_MEMORY_SCOPE_SYSTEM)) - call) < 0) {
        dead = timed_out(tmo, call, tstart, spins);
        __builtin_amdgcn_s_sleep(GSPIN);
      }
      TICKET(t0);
      asm volatile("s_nop 4\n\tbuffer_load_dword %0, %2, %3, 0 offen sc0 sc1\n\ts_waitcnt vmcnt(0)" : "=v"(SN[0]), "+v"(t0) : "v"(sgo), "s"(rs_my) : "memory");
    }
    t0 = rfl(t0);
    unsigned long long ent;
    asm volatile("ds_read_b64 %0, %1 offset:8192\n\ts_waitcnt lgkmcnt(0)" : "=v"(ent) : "v"(lds + (t0 & 0xff8)) : "memory");
    const unsigned ey = rfl(ent >> 32);
    so1 = rfl((unsigned)ent) + (t0 & 7) * 2048;
    do1 = (ey & ~0x3fffu) + (t0 & 7) * 2048;
    c1 = ey & 127;
    const bool late = t0 >= nfree;
    const unsigned long long want = late ? 0ull : (1ull << 32) | (1ull << ((ey >> 7) & 63));
    bool told = !push;
    while (!dead && (__ballot((int)(SN[0] - thg) >= 0) & want) != want) {
      if (!told) {
        told = true;
        asm volatile("s_nop 4\n\ts_waitcnt vmcnt(0)\n\tbuffer_atomic_add %0, %1, %2, %3 offen sc1" :: "v"(8u), "v"(l0off), "s"(rs_d), "s"(flag_off) : "memory");
      } else {
        dead = timed_out(tmo, call, tstart, spins);
        __builtin_amdgcn_s_sleep(PSLEEP);
      }
      SN[0] = __hip_atomic_load((gu32*)(myws + sgo), __ATOMIC_RELAXED, __HIP_MEMORY_SCOPE_SYSTEM);
      asm volatile("" : "+v"(SN[0]));
    }
    if (dead || late) { so1 = do1 = OOB; c1 = NONE; sv[1] = dead ? BIG : t0; }
    asm volatile("s_nop 4\n\tbuffer_load_dword %2, %7, %8, %9 offen sc0 sc1\n\t"
                 "buffer_atomic_add %3, %12, %8, %14 offen sc0\n\t"
                 "buffer_load_dwordx4 %0, %4, %5, %6 offen sc0 sc1\n\t"
                 "buffer_load_dwordx4 %1, %10, %5, %6 offen offset:1024 sc0 sc1\n\ts_waitcnt vmcnt(4)\n\t"
                 "buffer_atomic_add %11, %12, %5, %13 offen sc1"
                 : "=&v"(D[1][0]), "=&v"(D[1][1]), "=&v"(SN[1]), "+v"(TK[1])
                 : "v"(l16), "s"(rs_d), "s"(so1), "v"(l4), "s"(rs_my), "s"(snap_off), "v"((ey >> 13) & 1 ? l40 : l16), "v"(8u), "v"(l0off),
                   "s"(told ? OOB : flag_off), "s"(sv[1] == NOSV ? tick_off : OOB) : "memory");
#pragma unroll
    for (int i = 0; i < (PAIR ? 0 : 2); ++i)
      asm volatile("s_nop 4\n\tbuffer_store_dword %0, %0, %1, %2 offen" :: "v"(l4), "s"(rs_my), "s"(OOB) : "memory");
  }
  for (;;) {
    if (iter(std::integral_constant<int, 0>{})) break;
    if (iter(std::integral_constant<int, 1>{})) break;
  }
  wait_vm<0>();
  if (dead && lane == 0) p.out[obase / 4] = __builtin_nanf("");
}

__global__ void __launch_bounds__(512, 2) fused_kernel(Params p) {
  __shared__ __attribute__((aligned(16))) uint8_t smem_[LDS_BYTES];
  lu8* const smem = (lu8*)smem_;
  const int tid = threadIdx.x;
  const int lane = tid & 63;
  const int wave = __builtin_amdgcn_readfirstlane(tid >> 6);
  const int b = blockIdx.x;
  if (__builtin_expect(b < NSEND, 1)) {
    const int gw = b < NSEND - 1 ? wave * (NSEND - 1) + b : NSEND * 8 - 1 - wave;
    v4u X[4][2];
    unsigned xo = 0, call = 0;
    if (__builtin_expect(gw < 7 * NWL, 1)) {
      call = push_start(p, gw, lane, X, xo);
    } else {
      if (gw >= 7 * NPL && gw < NSEND * 8 - 2) return;
      if (lane == 0) call = __hip_atomic_fetch_add((gu32*)(p.ws[0] + OFF_CALL + gw * 16), 1u, __ATOMIC_RELAXED, __HIP_MEMORY_SCOPE_AGENT);
      call = rfl(call) + 1;
#pragma unroll
      for (int j = 0; j < 4; ++j) asm volatile("" : "=v"(X[j][0]), "=v"(X[j][1]));
    }
    const int rank = (int)rfl((unsigned)p.rank);
    gu8* const myws = rfl_ptr(p.ws[0]);
    const unsigned lds = (unsigned)(uintptr_t)smem + wave * 12288;
    if (gw == NSEND * 8 - 1) monitor(myws, rfl_ptr(p.ws[1]), rank, call, lane);
    else if (gw == NSEND * 8 - 2) planner(myws, call, lane);
    else if (gw % 7 == 0) puller<true>(p, myws, rank, gw, lane, lds, call, X, xo);
    else puller<false>(p, myws, rank, gw, lane, lds, call, X, xo);
    return;
  }
  const unsigned long long tstart = __builtin_amdgcn_s_memrealtime();
  const int rank = (int)rfl((unsigned)p.rank);
  gu8* const myws = rfl_ptr(p.ws[0]);
  gu32* const tmo = (gu32*)(myws + OFF_TMO);
  const int g = lane >> 4, r = lane & 15;
  volatile __attribute__((address_space(3))) unsigned* const bc =
      (volatile __attribute__((address_space(3))) unsigned*)(smem + 3 * LDS_STAGE);

  if (tid == 0) {
    const unsigned t = __hip_atomic_fetch_add((gu32*)(myws + OFF_CALL + b * 128), 1u, __ATOMIC_RELAXED, __HIP_MEMORY_SCOPE_AGENT);
    bc[0] = t + 1;
  }
  if (tid >= 64 && tid < 80) bc[64 + ((tid >> 2) & 3) * 64 + (tid & 3)] = 0;
  unsigned call;
  while (__builtin_amdgcn_s_memrealtime() - tstart < (unsigned)(PRE_T0 + (b >= 128 ? (b & 7) * PRE_DT : b >= 80 && b < 96 ? PRE_DT : 0))) __builtin_amdgcn_s_sleep(2);
  if (b >= 128 || b < 96) {
    const int jp = b < 96 ? (b >> 4) & 1 : b & 7, ksl = b < 96 ? b & 3 : (b >> 3) & 3, dq = b < 96 ? (b >> 2) & 3 : (b - 128) >> 5;
    const int cq = (rank >> 1) ^ dq;
    const bool hp = jp < 2;
    const unsigned m0 = transpose_strip(p.w + (size_t)(ksl * 128) * GN + cq * NQ + jp * 512 + wave * 64, smem + wave * 8192, lane);
    *(lv4u*)(smem + 65536 + tid * 16) = (v4u){0, 0, 0, 0};
    __syncthreads();
    call = rfl(bc[0]);
    const unsigned dbase = (dq == 0 ? OFF_STAGE + (call & 1) * STAGE_BYTES + rank * SRC_BYTES + SX_W
                                    : OFF_WT + (call & 1) * LOC_BYTES + cq * (2u << 20))
                           + (jp * 2 + (wave >> 2)) * 131072 + ksl * 32768 + (wave & 3) * 8192 + lane * 16;
    const __amdgpu_buffer_rsrc_t rw = __builtin_amdgcn_make_buffer_rsrc((void*)myws, 0, WS_NEED, 0x00020000);
    unsigned cnt = 1;
#pragma unroll 1
    for (int e = b < 96 ? 2 : 0; e < (hp && b >= 128 ? 2 : 4); ++e) {
      v4u a0 = *(const lv4u*)(smem + wave * 8192 + e * 2048 + lane * 16), a1 = *(const lv4u*)(smem + wave * 8192 + e * 2048 + 1024 + lane * 16);
      __builtin_amdgcn_raw_buffer_store_b128(a0, rw, dbase + e * 2048, 0, AUX_SYS);
      __builtin_amdgcn_raw_buffer_store_b128(a1, rw, dbase + e * 2048 + 1024, 0, AUX_SYS);
      const unsigned ex = (unsigned)(uintptr_t)smem + 65536 + wave * 1024 + (e >> 1) * 512;
      pack_piece(a0, a1, m0, ex, (e & 1) * 2048, cnt, lane);
      __builtin_amdgcn_raw_buffer_store_b128(a0, rw, dbase + PACKED + e * 2048, 0, AUX_SYS);
      __builtin_amdgcn_raw_buffer_store_b128(a1, rw, lane < 32 ? dbase + PACKED + e * 2048 + 1024 : 0x80000000u, 0, AUX_SYS);
      if (e & 1) {
        v4u l;
        asm volatile("ds_read_b128 %0, %1\n\ts_waitcnt lgkmcnt(0)" : "=v"(l) : "v"(ex + (lane & 15) * 16) : "memory");
        l[0] |= (lane == 0 ? m0 | (cnt > 128 ? 0x4000u : 0u) : 0u) | (cnt > 64 ? 0x8000u : 0u);
        __builtin_amdgcn_raw_buffer_store_b128(l, rw, lane < 16 ? dbase + PACKED + e * 2048 - 512 + (lane >> 3) * 1920 : 0x80000000u, 0, AUX_SYS);
        cnt = 1;
      } else if (cnt > 64) {
        cnt = 129;
      }
    }
    wait_vm<0>();
    __syncthreads();
    if (tid < 2) {
      __hip_atomic_fetch_add((gu32*)(myws + OFF_SNAP + (cq * 2 + tid) * 256) + 32 + jp, hp ? 1u : 2u, __ATOMIC_RELAXED, __HIP_MEMORY_SCOPE_AGENT);
    } else if (tid >= 64 && tid < 66) {
      __hip_atomic_fetch_add((gu32*)(p.ws[dq * 2 + tid - 64] + OFF_SNAP + rank * 256) + 40 + jp, hp ? 1u : 2u, __ATOMIC_RELAXED,
                             __HIP_MEMORY_SCOPE_SYSTEM);
    }
  } else {
    const int rg = b - 96, ksl = wave >> 1, hf = wave & 1;
    const uint8_t* src = p.x + (size_t)(rg * 128 + hf * 64 + r) * KSH + ksl * 128 + g * 16;
    v4u B[8];
#pragma unroll
    for (int e = 0; e < 8; ++e) B[e] = *(const v4u*)(src + (e >> 1) * 16 * KSH + (e & 1) * 64);
    __syncthreads();
    call = rfl(bc[0]);
    const int d = (rg >> 2) ^ rank;
    const bool loc = d == 0 || (d > 1 && ((rg >> 1) & 1) == (rank & 1));
    const unsigned woff = wave * 8192 + lane * 16;
    const unsigned dloc = OFF_STAGE + (call & 1) * STAGE_BYTES + rank * SRC_BYTES
                          + (d == 0 ? SX_O + (rg & 3) * 65536 : (d - 2) * 131072 + (rg & 1) * 65536) + woff;
    const __amdgpu_buffer_rsrc_t rw = __builtin_amdgcn_make_buffer_rsrc((void*)myws, 0, WS_NEED, 0x00020000);
    if (d != 0) {
#pragma unroll
      for (int e = 0; e < 8; ++e) __builtin_amdgcn_raw_buffer_store_b128(B[e], rw, OFF_XT + (call & 1) * LOC_BYTES + rg * 65536 + woff + e * 1024, 0, AUX_SYS);
    }
    if (loc) {
#pragma unroll
      for (int e = 0; e < 8; ++e) __builtin_amdgcn_raw_buffer_store_b128(B[e], rw, dloc + e * 1024, 0, AUX_SC1);
    }
    asm volatile("s_waitcnt vmcnt(0)" ::: "memory");
    __syncthreads();
    if (tid == 0) {
      __hip_atomic_store((gu32*)(myws + OFF_XTF) + rg, call, __ATOMIC_RELAXED, __HIP_MEMORY_SCOPE_AGENT);
    } else if (tid > 64 && tid < 72 && d != 0) {
      const int e = tid - 64;
      if (e == d || ((e ^ d) > 1 && ((rank ^ e) & 1) == ((rg >> 1) & 1) && ((rg & 1) || (e == 1 && d == 6))))
        __hip_atomic_fetch_add((gu32*)(p.ws[e] + OFF_SNAP + rank * 256) + 48, 1u, __ATOMIC_RELAXED, __HIP_MEMORY_SCOPE_SYSTEM);
    }
  }
  const int nb = b & 7, u = (b - NSEND) >> 3;
  const int wm = wave >> 2, wn = wave & 3;
  const float scale = p.sx[0] * p.sw[0];
  const unsigned o_stg = OFF_STAGE + (call & 1) * STAGE_BYTES, voff = wave * 1024 + lane * 16, moff = lane < 4 ? lane * 4 : 0x40000000u;
  const __amdgpu_buffer_rsrc_t rws = __builtin_amdgcn_make_buffer_rsrc((void*)myws, 0, WS_NEED, 0x00020000);
  gu32* const fptr = (gu32*)(myws + OFF_MASK + b * 128);

#pragma unroll 1
  for (int pass = u < 12 ? (nb >= 5 ? 0 : 3) : nb < 4 ? 1 : 0; pass < 5; pass = u < 12 && pass == 0 ? 3 : pass + 1) {
    const int k2 = (u - 8) * 8 + nb;
    const bool mini = pass == 1 || pass == 2, own = pass == 4 && (u < 8 || (k2 >= 24 && u < 12));
    if (pass == 4 && u >= 12 && nb >= H12) break;
    const int v = u >= 12 ? u - 12 : pass == 4 ? k2 % 12 : u;
    const int peer = own ? rank : rank ^ (2 + (v >> 1));
    const int rb = own ? (u < 8 ? u : nb) & 3 : pass == 0 && u < 12 ? 0 : v & 1;
    const int blk = pass == 0 ? (u < 12 ? 0 : nb - 4) : mini ? 2 * nb + pass - 1 : own ? (u < 8 ? nb + (u >> 2) * 8 : 14 + (nb >> 2)) : pass == 4 ? (u >= 12 ? 12 + nb : 14 + k2 / 12)
                    : u >= 12 ? nb + 8 : nb;
    const bool half = !mini && (((blk < 4 || (blk >= 12 && blk < 12 + H12)) && !own) || blk >= 14);
    const int hsel = pass == 0 || (pass == 4 && u >= 8);
    const bool eighth = half && blk == 0 && rb == 0;

    const unsigned cf = lane < 4 || lane >= 28 ? 0u : (unsigned)(lane - 4) >> 2;
    const unsigned tab = (lane < 4 ? rank : lane < 28 ? rank ^ (2 + cf) : rank ^ 1) | ((lane & 3) << 3);
    unsigned todo = mini ? 15u : (unsigned)__ballot(own || (tab & 7) != (unsigned)peer);
    unsigned need = 31;
    bool planned = false;
    auto hasplan = [&](unsigned val) __attribute__((always_inline)) {
      const unsigned plan = __builtin_amdgcn_readlane(val, 2);
      return (plan >> 30) == (call & 3) && (plan & 31) >= NMIN;
    };
    auto replan = [&](unsigned val) __attribute__((always_inline)) {
      planned = hasplan(val);
      const unsigned plan = planned ? __builtin_amdgcn_readlane(val, 2) : NMIN * 0x2108421u;
      const unsigned ks = lane & 3, i = ilv(2 * blk + (ks >> 1)), n = plan_n(plan, cf), r = plan_r(plan);
      const unsigned pos = plan_pos(plan, i, cf);
      const bool par = lane >= 28, dir = i < n || (ks & 1) == (unsigned)(rank & 1);
      const unsigned rel = pos + NFU >= r ? C_RF + pos + NFU - r + 1 : rcls(pos / 6) + 1;
      const unsigned nxx = rb == 1 && ks >= XK + XF ? C_X1 + 1 : 1u;
      const unsigned nf = par ? 0u : i >= n && !planned ? 31u : own ? C_O + 1 : max(nxx, dir ? 1 + wcls(i < n ? 2 * i + (ks & 1) : n + i) : 1u);
      const unsigned np = par ? (own ? C_XO + rb * 4 + ks + 1 : max(nxx, 1 + pcls(blk * 4 + ks))) : dir ? 0u : rel;
      const unsigned nr = par ? (own ? C_XO - 8 + rb * 2 : np - 2) : dir ? 0u
                              : pos + NFU >= r + 8 ? rel - 8 : rcls((pos - min(pos, 5u)) / 6);
      need = nf | (np << 8) | (nr << 16);
    };
    auto arrived = [&](unsigned val, bool soon) __attribute__((always_inline)) {
      unsigned wa = __builtin_amdgcn_readlane(val, 0), wb = __builtin_amdgcn_readlane(val, 1);
      wa = (wa >> 30) == (call & 3) ? wa : 0u;
      wb = (wb >> 16) == (call & 0xffffu) ? wb & 0xff : 0u;
      const unsigned pf = (wa >> (5 * cf)) & 31;
      return (unsigned)__ballot(pf + soon >= (need & 0xff) && wb >= (soon ? need >> 16 : (need >> 8) & 0xff)) & todo;
    };
    if (wave == 0) {
      const int rg = (mini || own ? rank : peer) * 4 + (own ? 0 : ((mini ? peer : rank) & 1) * 2) + rb;
      gu32* f = lane < 4 ? fptr + lane : lane == 4 ? (gu32*)(myws + OFF_SNAP + (mini ? peer : rank) * 256) + 32 + (blk >> 1)
                                                   : (gu32*)(myws + OFF_XTF) + rg;
      const unsigned want = lane == 4 ? call * 8 : call;
      if (mini)
        while (__builtin_amdgcn_s_memrealtime() - tstart < (unsigned)((nb < 4 ? MINI_T0 : MINI_T1) + MINI_DT * ((pass - 1) * 48 + (nb & 3) * 12 + v)))
          __builtin_amdgcn_s_sleep(8);
      const unsigned long long t0 = __builtin_amdgcn_s_memrealtime();
      unsigned spins = 63, val;
      for (;;) {
        val = __hip_atomic_load(f, __ATOMIC_RELAXED, __HIP_MEMORY_SCOPE_AGENT);
        if (__all(lane < 4 || val == want)) break;
        if (timed_out(tmo, call, t0, spins)) { val = 1; break; }
        __builtin_amdgcn_s_sleep(LSLEEP);
      }
      if (lane < 4) bc[4 + lane] = val;
    }
    __syncthreads();
    bool ok = true;

    v4f acc[4][4];
#pragma unroll
    for (int i = 0; i < 4; ++i)
#pragma unroll
      for (int j = 0; j < 4; ++j) acc[i][j] = (v4f){0.f, 0.f, 0.f, 0.f};
    const int r16 = half ? (wave >> 1) * 2 : wm * 4;
    const int strip = half ? wave & 1 : wn;

    auto kloop = [&](auto hf) {
      constexpr bool HALF = decltype(hf)::value;
      constexpr int NI = HALF ? 2 : 4, L = 2 + NI, D = HALF ? 3 : 2;
      constexpr int SB = L * 8192, SE = (D + 1) * SB;
      const unsigned xa = o_stg + (mini ? SX_O + ((peer & 1) * 2 + rb) * 65536 : own ? SX_O + rb * 65536 : (v >> 1) * 131072 + rb * 65536);
      const unsigned wb = (mini ? OFF_WT + (call & 1) * LOC_BYTES + (peer >> 1) * (2u << 20) - rank * SRC_BYTES : o_stg + SX_W)
                          + blk * 131072 + (HALF ? hsel * 16384 : 0);
      const int boff = (r & 1) * 1024 + 16 * ((r >> 1) + 8 * ((g & 1) ^ (r & 1)) + 16 * (g >> 1));
      const lu8* const la0 = smem + r16 * 2048 + lane * 16;
      const lu8* const lb0 = smem + 16384 + strip * 8192 + boff;
      int si = 0, sc = SB;
      unsigned ready = 15, live = 0, val = 0;
      todo &= ~15u;
      bool idle = true, slow = true;
      v8i af[NI], bf[4];
      auto readf = [&]() {
#pragma unroll
        for (int i = 0; i < 4; ++i) {
          if (i < NI) {
            const v4i lo = *(const lv4i*)(la0 + sc + i * 2048), hi = *(const lv4i*)(la0 + sc + i * 2048 + 1024);
            af[i] = __builtin_shufflevector(lo, hi, 0, 1, 2, 3, 4, 5, 6, 7);
          }
          const v4i lo = *(const lv4i*)(lb0 + sc + i * 2048), hi = *(const lv4i*)(lb0 + sc + i * 2048 + 512);
          bf[i] = __builtin_shufflevector(lo, hi, 0, 1, 2, 3, 4, 5, 6, 7);
        }
        __builtin_amdgcn_sched_barrier(0);
      };
      auto issue = [&](bool real) {
        const unsigned e = __builtin_amdgcn_readlane(tab, __builtin_ctz(ready | 0x80000000u));
        const unsigned so = real ? (e & 7) * SRC_BYTES : 0x80000000u, js = (e >> 3) & 3;
        lu8* l = smem + si + wave * 1024;
#pragma unroll
        for (int i = 0; i < 2; ++i)
          __builtin_amdgcn_raw_ptr_buffer_load_lds(rws, l + i * 8192, 16, voff, xa + so + js * 16384u + i * 8192, 0, AUX_SYS);
#pragma unroll
        for (int i = 0; i < NI; ++i)
          __builtin_amdgcn_raw_ptr_buffer_load_lds(rws, l + 16384 + i * 8192, 16, voff, wb + so + js * 32768u + i * 8192, 0, AUX_SYS);
        ready &= ready - 1;
        si = sc;
        sc = sc + SB == SE ? 0 : sc + SB;
      };
      auto fetch = [&](unsigned slot) {
        __builtin_amdgcn_raw_ptr_buffer_load_lds(rws, smem + LDS_MB + (wave == 0 ? (slot & 3) * 256 : 1024), 4, moff,
                                                 wave == 0 && todo != 0 ? OFF_MASK + b * 128 : 0x80000000u, 0, AUX_SC1);
      };
      auto mma = [&]() {
        __builtin_amdgcn_s_setprio(1);
#pragma unroll
        for (int i = 0; i < NI; ++i) {
#pragma unroll
          for (int j = 0; j < 4; ++j)
            acc[i][j] = __builtin_amdgcn_mfma_scale_f32_16x16x128_f8f6f4(af[i], bf[j], acc[i][j],   0,   1,
                                                                          0, 0x7f7f7f7f, 0, 0x7f7f7f7f);
          asm volatile("" : "+v"(acc[i][0]), "+v"(acc[i][1]), "+v"(acc[i][2]), "+v"(acc[i][3]));
        }
        __builtin_amdgcn_s_setprio(0);
      };
#pragma unroll
      for (int i = 0; i < D; ++i) issue(true);
      live = (2u << D) - 2;
      wait_vm<(D - 1) * L>();
      if (D == 3) fetch(3);
      __builtin_amdgcn_s_barrier();
      if (wm == 1) __builtin_amdgcn_s_barrier();
      for (unsigned it = 0;; ++it) {
        if (__builtin_expect(slow, 0)) {
          if (idle) val = bc[4 + (it & 1) * 4 + (lane & 3)];
          if (!planned && todo != 0 && (it == 0 || hasplan(val))) replan(val);
          unsigned m = arrived(val, false);
          if (__builtin_amdgcn_readlane(val, 3) != 0) { ok = false; m = todo; }
          ready |= m;
          todo &= ~m;
        }
        int n = __builtin_popcount(ready) - (todo != 0 ? 2 : 1);
        if (n > 0) {
          live = n >= D ? (2u << D) - 2 : (live >> n) | ((2u << D) - (2u << (D - n)));
          do {
            __builtin_amdgcn_sched_barrier(0);
            readf();
            issue(true);
            wait_vm<(D - 1) * L>();
            raw_barrier();
            __builtin_amdgcn_sched_barrier(0);
            mma();
            __builtin_amdgcn_sched_barrier(0);
            __builtin_amdgcn_s_barrier();
            asm volatile("" ::: "memory");
          } while (--n > 0);
          if (D == 3) fetch(it + 3);
        }
        __builtin_amdgcn_sched_barrier(0);
        readf();
        fetch(it);
        live = (live >> 1) | (ready != 0 ? 1u << D : 0u);
        const bool dry = ready == 0;
        issue(ready != 0);
        if (dry) wait_vm<L>();
        wait_vm<(D - 1) * (L + 1)>();
        raw_barrier();
        __builtin_amdgcn_sched_barrier(0);
        const bool last = (ready | todo) == 0;
        val = bc[(LDS_MB - 3 * LDS_STAGE) / 4 + ((it + (dry ? 0 : 5 - D)) & 3) * 64 + (lane & 3)];
        __builtin_amdgcn_s_setprio(1);
#pragma unroll
        for (int i = 0; i < NI; ++i)
#pragma unroll
          for (int j = 0; j < 4; ++j)
            acc[i][j] = __builtin_amdgcn_mfma_scale_f32_16x16x128_f8f6f4(af[i], bf[j], acc[i][j],   0,   1,
                                                                          0, 0x7f7f7f7f, 0, 0x7f7f7f7f);
        const unsigned m = arrived(val, false);
        ready |= m;
        todo &= ~m;
        idle = (ready | (live >> 1)) == 0;
        slow = idle | (!planned & hasplan(val));
#pragma unroll
        for (int i = 0; i < 3; ++i) __builtin_amdgcn_sched_group_barrier(0x008, 1, 0);
#pragma unroll
        for (int i = 3; i < 4 * NI; ++i) {
          __builtin_amdgcn_sched_group_barrier(0x006, HALF ? 8 : 4, 0);
          __builtin_amdgcn_sched_group_barrier(0x008, 1, 0);
        }
#pragma unroll
        for (int i = 0; i < NI; ++i) asm volatile("" : "+v"(acc[i][0]), "+v"(acc[i][1]), "+v"(acc[i][2]), "+v"(acc[i][3]));
        __builtin_amdgcn_s_setprio(0);
        __builtin_amdgcn_sched_barrier(0);
        if (last) break;
        if (__builtin_expect(idle, 0) && wave == 0) {
          const unsigned long long t0 = __builtin_amdgcn_s_memrealtime();
          unsigned spins = 63;
          for (;;) {
            val = __hip_atomic_load(fptr + (lane & 3), __ATOMIC_RELAXED, __HIP_MEMORY_SCOPE_AGENT);
            if (arrived(val, false) != 0 || (!planned && hasplan(val))) break;
            if (timed_out(tmo, call, t0, spins)) { val = 1; break; }
            if (arrived(val, true) == 0) __builtin_amdgcn_s_sleep(32);
          }
          if (lane < 4) bc[4 + (~it & 1) * 4 + lane] = val;
        }
        raw_barrier();
      }
      if (wm == 0) __builtin_amdgcn_s_barrier();
#pragma unroll 1
      for (int k = D - 1; k >= 0; --k) {
        if (k == 0) wait_vm<0>(); else if (D == 2 || k == 1) wait_vm<L>();
        __builtin_amdgcn_s_barrier();
        asm volatile("" ::: "memory");
        live >>= 1;
        if (live & 1) { readf(); mma(); }
        sc = sc + SB == SE ? 0 : sc + SB;
      }
    };
    if (eighth) {
      const int rh = u < 12 ? (pass == 0 ? u & 1 : 0) : 1, s8 = u < 12 && pass == 0 ? nb - 4 : 0;
      const int boff = (r & 1) * 1024 + 16 * ((r >> 1) + 8 * ((g & 1) ^ (r & 1)) + 16 * (g >> 1));
      const unsigned so = o_stg + (__builtin_amdgcn_readlane(tab, wave * 4) & 7) * SRC_BYTES;
      const unsigned xa = so + (v >> 1) * 131072 + rh * 8192, wb = so + SX_W + s8 * 8192;
      ok = rfl(bc[7]) == 0;
      replan(0);
#pragma unroll 1
      for (int h = 0; h < 2 && ((todo >> (wave * 4)) & 1); ++h) {
        const unsigned long long t0 = __builtin_amdgcn_s_memrealtime();
        unsigned spins = 63;
        while (wave != 0) {
          const unsigned val = __hip_atomic_load(fptr + (lane & 3), __ATOMIC_RELAXED, __HIP_MEMORY_SCOPE_AGENT);
          if (((arrived(val, false) >> (wave * 4 + h * 2)) & 3) == 3) break;
          if (timed_out(tmo, call, t0, spins)) { ok = false; break; }
          if (((arrived(val, true) >> (wave * 4 + h * 2)) & 3) != 3) __builtin_amdgcn_s_sleep(32);
        }
        v4i fa[2][4][2], fb[2][4][2];
#pragma unroll
        for (int k = 0; k < 2; ++k)
#pragma unroll
          for (int i = 0; i < 4; ++i)
#pragma unroll
            for (int e = 0; e < 2; ++e) {
              fa[k][i][e] = __builtin_bit_cast(v4i, __builtin_amdgcn_raw_buffer_load_b128(rws, lane * 16 + (i & 1) * 2048 + e * 1024, xa + (h * 2 + k) * 16384 + (i >> 1) * 4096, AUX_SYS));
              fb[k][i][e] = __builtin_bit_cast(v4i, __builtin_amdgcn_raw_buffer_load_b128(rws, boff + (i & 1) * 2048 + e * 512, wb + (h * 2 + k) * 32768 + (i >> 1) * 4096, AUX_SYS));
            }
        __builtin_amdgcn_sched_barrier(0);
#pragma unroll
        for (int k = 0; k < 2; ++k)
#pragma unroll
          for (int i = 0; i < 4; ++i)
#pragma unroll
            for (int j = 0; j < 4; ++j)
              acc[i][j] = __builtin_amdgcn_mfma_scale_f32_16x16x128_f8f6f4(
                  __builtin_shufflevector(fa[k][i][0], fa[k][i][1], 0, 1, 2, 3, 4, 5, 6, 7),
                  __builtin_shufflevector(fb[k][j][0], fb[k][j][1], 0, 1, 2, 3, 4, 5, 6, 7), acc[i][j], 0, 1, 0, 0x7f7f7f7f, 0, 0x7f7f7f7f);
      }
      const float sc = ok ? 1.f : __builtin_nanf("");
#pragma unroll
      for (int e = 0; e < 16; ++e)
        *(__attribute__((address_space(3))) v4f*)(smem + wave * 16384 + e * 1024 + lane * 16) =
            (v4f){acc[e >> 2][0][e & 3] * sc, acc[e >> 2][1][e & 3], acc[e >> 2][2][e & 3], acc[e >> 2][3][e & 3]};
      __syncthreads();
#pragma unroll
      for (int e = 0; e < 2; ++e) {
        v4f o = {0.f, 0.f, 0.f, 0.f};
#pragma unroll
        for (int w = 0; w < 8; ++w) o += *(const __attribute__((address_space(3))) v4f*)(smem + w * 16384 + (wave * 2 + e) * 1024 + lane * 16);
        __builtin_amdgcn_raw_buffer_store_b128(__builtin_bit_cast(v4u, o), rws, (g * 4 + e) * 1024 + r * 16 +
            OFF_OST + (call & 1) * OST_BYTES + peer * (4u << 20) + (rh * 64 + (wave >> 1) * 16 + (wave & 1) * 2) * 1024 + s8 * 256, 0, AUX_SYS);
      }
      wait_vm<0>();
      __syncthreads();
      if (tid == 0)
        __hip_atomic_fetch_add((gu32*)(p.ws[peer ^ rank] + OFF_SNAP + rank * 256), 1u, __ATOMIC_RELAXED, __HIP_MEMORY_SCOPE_SYSTEM);
      continue;
    }
    if (half) kloop(std::true_type{}); else kloop(std::false_type{});

    {
      const bool toout = mini || own;
      const __amdgpu_buffer_rsrc_t ro = toout
          ? __builtin_amdgcn_make_buffer_rsrc(p.out, 0, OUT_BYTES, 0x00020000)
          : __builtin_amdgcn_make_buffer_rsrc((void*)(myws + OFF_OST + (call & 1) * OST_BYTES), 0, OST_BYTES, 0x00020000);
      const unsigned ostr = toout ? (unsigned)(GN * 4) : 1024u;
      const int orb = mini ? (peer & 1) * 2 + rb : rb;
      const unsigned obase = (toout ? (unsigned)((orb * 128 * GN + (peer >> 1) * NQ + blk * 256) * 4)
                                    : (unsigned)((peer * 32 + blk * 2 + rb) * (128 << 10)))
                             + (half ? hsel * 512 : 0) + (unsigned)(r16 * 16 + g * 4) * ostr + strip * 256 + r * 16;
      const float sc = !ok ? __builtin_nanf("") : (own ? scale : 1.f);
#pragma unroll
      for (int i = 0; i < 4; ++i)
        if (i < 2 || !half) {
#pragma unroll
          for (int reg = 0; reg < 4; ++reg) {
            v4f o = {acc[i][0][reg] * sc, acc[i][1][reg] * sc, acc[i][2][reg] * sc, acc[i][3][reg] * sc};
            if (own) o = (v4f){silu(o[0]), silu(o[1]), silu(o[2]), silu(o[3])};
            __builtin_amdgcn_raw_buffer_store_b128(__builtin_bit_cast(v4u, o), ro, obase + (unsigned)(i * 16 + reg) * ostr, 0, AUX_SYS);
          }
        }
      wait_vm<0>();
      __syncthreads();
      if (tid == 0) {
        if (mini)
          __hip_atomic_fetch_add((gu32*)(myws + OFF_SNAP + peer * 256) + 60 + (blk >> 2), 1u, __ATOMIC_RELAXED, __HIP_MEMORY_SCOPE_AGENT);
        else if (!own)
          __hip_atomic_fetch_add((gu32*)(p.ws[peer ^ rank] + OFF_SNAP + rank * 256) + blk * 2 + rb, half ? 4u : 8u, __ATOMIC_RELAXED,
                                 __HIP_MEMORY_SCOPE_SYSTEM);
      }
    }
  }
}

}

extern "C" void kernel_launch(int rank, int nranks, ncclComm_t comm,
                              hipStream_t stream,
                              void* const* d_in, const int* in_sizes, int n_in,
                              void* d_out, int out_size,
                              const int* in_dtypes, int out_dtype,
                              void* const* d_out_all,
                              void* const* d_ws_all, size_t ws_bytes) {
  (void)comm; (void)in_dtypes; (void)out_dtype; (void)d_out_all;
  if (nranks != NR || n_in != 4 || ws_bytes < (size_t)WS_NEED || out_size != 512 * GN ||
      in_sizes[0] != 4096 * KSH || in_sizes[1] != KSH * GN || (unsigned)rank >= (unsigned)NR) {
    fprintf(stderr, "kernel_launch: unexpected problem configuration\n");
    return;
  }
  Params p;
  p.x = (const uint8_t*)d_in[0];
  p.w = (const uint8_t*)d_in[1];
  p.sx = (const float*)d_in[2];
  p.sw = (const float*)d_in[3];
  p.out = (float*)d_out;
  for (int i = 0; i < NR; ++i) p.ws[i] = (uint8_t*)d_ws_all[rank ^ i];
  p.rank = rank;
  hipLaunchKernelGGL(fused_kernel, dim3(NWG), dim3(512), 0, stream, p);
  HIP_CHECK(hipGetLastError());
}
```
